```python
import numpy as np

kernel_name = "gpu_gemm_rs_m2048_k2048_n2048_f32_silu"


ABSMAX_RTOL = 0.001

COMPUTE_DTYPE = "float32"
EPILOGUE = "silu"
IN_QUANT = None
OUT_QUANT = None
IN_DTYPES = {"x": "float32", "w_mat": "float32"}
WORLD_SIZE = 8

def setup_inputs():

    x = np.random.randn(2048, 2048).astype(np.float32)
    w_mat = (np.random.randn(2048, 2048) / np.sqrt(2048)).astype(np.float32)
    return {"x": x, "w_mat": w_mat}

def _epilogue(y):
    return (y / (1.0 + np.exp(-np.clip(y, -60.0, 60.0)))).astype(np.float32)

def reference(x, w_mat):
    y = (x @ w_mat).astype(np.float32)
    return _epilogue(y)
```

```cpp
#include <hip/hip_runtime.h>
#include <rccl/rccl.h>
#include <cstdio>

typedef _Float16 h8 __attribute__((ext_vector_type(8)));
typedef float f4 __attribute__((ext_vector_type(4)));
typedef unsigned u4 __attribute__((ext_vector_type(4)));
typedef int i4 __attribute__((ext_vector_type(4)));

constexpr int NR = 8;
constexpr int GN = 2048;
constexpr int KS = 256;
constexpr int NWG = 256;
constexpr int NW = 8;
constexpr int NTHREADS = NW * 64;

#define SPAD 9
constexpr unsigned WB_SRC = (512u + SPAD) << 10, XA_SRC = (256u + SPAD) << 10;
constexpr unsigned WB_PAR = 8 * WB_SRC;
constexpr unsigned XA_PAR = 8 * XA_SRC;
constexpr unsigned WB_OFF = 0;
constexpr unsigned XA_OFF = WB_OFF + 2 * WB_PAR;
constexpr unsigned EW_OFF = XA_OFF + 2 * XA_PAR;
constexpr unsigned EX_OFF = EW_OFF + 65536;
constexpr unsigned CTR_OFF = EX_OFF + 32768;
constexpr unsigned TE_OFF = CTR_OFF + 32768;
constexpr unsigned TMO_OFF = TE_OFF + 1024;
constexpr unsigned WS_USED = TMO_OFF + 128;
constexpr unsigned OUT_BYTES = 256u * GN * 4u;
constexpr unsigned IN_BYTES = 2048u * KS * 4u;
#define DEADLINE 100000000ull
constexpr unsigned long long DEADLINE_TICKS = DEADLINE;
constexpr int AUX_SYS = 17;
#define POLL_SLEEP 8
#define SLOT256 436
#define OSLOT256 SLOT256
constexpr unsigned EMPTY = 0xffffffffu, QNAN = 0x7fc00000u;
#define FAST_SLEEP 2
#define TE_PI 0
#define LAND256 25600
#define MARGIN256 23000
#define AGUESS 3
#define SPEC_FROM 6
#define OWN_AUX 17
#define OGAP256 0

struct Args {
  const float* x;
  const float* w;
  char* me;
  int rank;
  char* ws[NR];
  float* out[NR];
};

#define WAIT_VM0() asm volatile("s_waitcnt vmcnt(0)" ::: "memory")
#define WAIT_LGKM0() asm volatile("s_waitcnt lgkmcnt(0)" ::: "memory")
#define COMPILER_FENCE() asm volatile("" ::: "memory")

__device__ __forceinline__ __amdgpu_buffer_rsrc_t mk_rsrc(const void* p, unsigned bytes) {
  return __builtin_amdgcn_make_buffer_rsrc((void*)p, (short)0, (int)bytes, 0x00020000);
}
__device__ __forceinline__ void st128(__amdgpu_buffer_rsrc_t r, unsigned voff, u4 v) {
  __builtin_amdgcn_raw_buffer_store_b128(v, r, (int)voff, 0, AUX_SYS);
  __builtin_amdgcn_sched_barrier(0);
}
__device__ __forceinline__ void st32(__amdgpu_buffer_rsrc_t r, unsigned voff, unsigned soff, unsigned v) {
  __builtin_amdgcn_raw_buffer_store_b32(v, r, (int)voff, (int)soff, AUX_SYS);
}
__device__ __forceinline__ u4 ld128(__amdgpu_buffer_rsrc_t r, unsigned voff, unsigned soff) {
  return __builtin_amdgcn_raw_buffer_load_b128(r, (int)voff, (int)soff, AUX_SYS);
}
__device__ __forceinline__ unsigned ld32(__amdgpu_buffer_rsrc_t r, unsigned voff, unsigned soff) {
  return __builtin_amdgcn_raw_buffer_load_b32(r, (int)voff, (int)soff, AUX_SYS);
}
__device__ __forceinline__ float max3abs(float m, float a, float b) {
  float r;
  asm("v_max3_f32 %0, |%1|, |%2|, %3" : "=v"(r) : "v"(a), "v"(b), "v"(m));
  return r;
}
__device__ __forceinline__ int exp_of(unsigned amax_bits) { return (int)(amax_bits >> 23) - (127 + 14); }
__device__ __forceinline__ u4 pack8(const float (&v)[8], int ne, unsigned tag) {
  h8 h;
  h[0] = (_Float16)0.f;
#pragma unroll
  for (int j = 1; j < 8; ++j) h[j] = (_Float16)__builtin_amdgcn_ldexpf(v[j], ne);
  u4 r = __builtin_bit_cast(u4, h);
  const unsigned t = __builtin_bit_cast(unsigned, __builtin_amdgcn_cvt_pkrtz(__builtin_amdgcn_ldexpf(v[0], ne), 0.f));
  unsigned mag = t & 0x7fffu;
  mag = mag < 0x7ffeu ? mag : 0x7ffeu;
  mag += (mag ^ tag) & 1u;
  r.x |= (t & 0x8000u) | mag;
  return r;
}
__device__ __forceinline__ unsigned now256() { return (unsigned)__builtin_amdgcn_s_memrealtime() << 8; }
__device__ __forceinline__ void wait_until(unsigned t256) {
  int d = (int)(t256 - now256());
  if (d > (1 << 22)) return;
  while (d > 4096) { __builtin_amdgcn_s_sleep(2); d = (int)(t256 - now256()); }
  while (d > 0) d = (int)(t256 - now256());
}

struct Waiter {
  unsigned* tmo; unsigned call; unsigned long long t0; bool gave_up;
  __device__ __forceinline__ void start() { t0 = __builtin_amdgcn_s_memrealtime(); gave_up = false; }
  __device__ __forceinline__ bool expired() { return expired_t<POLL_SLEEP>(); }
  __device__ __forceinline__ bool expired_fast() { return expired_t<FAST_SLEEP>(); }
  template <int SLEEP> __device__ __forceinline__ bool expired_t() {
    const unsigned long long dt = __builtin_amdgcn_s_memrealtime() - t0;
    if (dt > 4000) {
      if (dt > DEADLINE_TICKS || __hip_atomic_load(tmo, __ATOMIC_RELAXED, __HIP_MEMORY_SCOPE_AGENT) == call) {
        __hip_atomic_store(tmo, call, __ATOMIC_RELAXED, __HIP_MEMORY_SCOPE_AGENT);
        gave_up = true;
        return true;
      }
    }
    __builtin_amdgcn_s_sleep(SLEEP);
    asm volatile("" ::: "memory");
    return false;
  }
};

#define STAMP(j) do {} while (0)

constexpr int LDS_PART = 0;
constexpr int LDS_SC = NW * 8192;
constexpr int LDS_XO = LDS_SC + 4096;
constexpr int LDS_CTL = LDS_XO + 32768;
constexpr int LDS_BYTES = LDS_CTL + 512;
constexpr int C_ITEM = 96, C_ST = 104, C_DEAD = 105, C_GRID = 106;

__device__ __forceinline__ void lds_arrive(unsigned* c, int lane) {
  WAIT_LGKM0();
  if (lane == 0) __hip_atomic_fetch_add(c, 1u, __ATOMIC_RELAXED, __HIP_MEMORY_SCOPE_WORKGROUP);
}
__device__ __forceinline__ void lds_wait(unsigned* c, unsigned n) {
  while (__hip_atomic_load(c, __ATOMIC_RELAXED, __HIP_MEMORY_SCOPE_WORKGROUP) < n) __builtin_amdgcn_s_sleep(1);
  COMPILER_FENCE();
}

__global__ void __launch_bounds__(NTHREADS) fused_gemm_rs_silu(Args a) {
  __shared__ __attribute__((aligned(16))) char smem[LDS_BYTES];
  unsigned* sctl = (unsigned*)(smem + LDS_CTL);
  const int b = blockIdx.x;
  const int lane = threadIdx.x & 63;
  const int wave = __builtin_amdgcn_readfirstlane(threadIdx.x >> 6);
  const int rank = a.rank;
  char* me = a.me;
  const int x8 = b & 7, ii = b >> 3, pi = ii & 15;
  const int l15 = lane & 15, kg = lane >> 4;
  const unsigned l16 = lane * 16;
  unsigned* ctr = (unsigned*)(me + CTR_OFF) + b * 32;
  const bool foreign = ii >= 16;
  const int ph = pi & 3;
  const int panel = ph * 4 + (x8 >> 1);
  const int rb = (foreign ? 0 : 8) + 4 * (x8 & 1) + (pi >> 2);
  STAMP(0);
  const int src = wave;
  const __amdgpu_buffer_rsrc_t r_me = mk_rsrc(me, WS_USED);
  const __amdgpu_buffer_rsrc_t r_out = mk_rsrc(a.out[rank & 7], OUT_BYTES);
  const bool cross = !foreign && ((x8 ^ rank) & 4);
  const bool late = foreign && ph == 0;
  auto mark_of = [&](int ln) {
    return (unsigned)(((64 * (x8 & 3) + 16 * (pi & 3) + 2 * wave + ((ln >> 4) & 1)) * GN + (x8 >> 2) * 1024 +
                       ((ln >> 5) * 4 + (pi >> 2)) * 64 + (ln & 15) * 4) * 4);
  };
  unsigned call, par, tag, cw;
  auto set_call = [&](unsigned call_v) {
    call = __builtin_amdgcn_readfirstlane(call_v) + 1u;
    par = call & 1u; tag = ((call + 1u) >> 1) & 1u;
    cw = call << 10;
  };
  const unsigned dst = (unsigned)x8;
  const __amdgpu_buffer_rsrc_t r_x = mk_rsrc(a.x, IN_BYTES);
  const unsigned xo = (unsigned)(l15 * KS + 32 * wave + 4 * kg) * 4u;
  auto load_x = [&](int row, float (&d)[8]) {
    const f4 lo = __builtin_bit_cast(f4, __builtin_amdgcn_raw_buffer_load_b128(r_x, xo, row * KS * 4, 0));
    const f4 hi = __builtin_bit_cast(f4, __builtin_amdgcn_raw_buffer_load_b128(r_x, xo + 64, row * KS * 4, 0));
    d[0] = lo.x; d[1] = lo.y; d[2] = lo.z; d[3] = lo.w; d[4] = hi.x; d[5] = hi.y; d[6] = hi.z; d[7] = hi.w;
  };
  auto max_only = [&](int it, const float (&d)[8]) {
    float m = max3abs(0.f, d[0], d[1]);
    m = max3abs(m, d[2], d[3]); m = max3abs(m, d[4], d[5]); m = max3abs(m, d[6], d[7]);
    __hip_atomic_fetch_max(&sctl[16 * it + l15], __float_as_uint(m), __ATOMIC_RELAXED, __HIP_MEMORY_SCOPE_WORKGROUP);
  };

  if (!foreign) {
    const __amdgpu_buffer_rsrc_t r_w = mk_rsrc(a.w, IN_BYTES);
    const int row_f = 256 * ((((int)(dst >> 2) ^ 1) << 2) | (pi >> 2)) + 64 * (int)(dst & 3) + 16 * (pi & 3);
    const unsigned wo = (unsigned)((32 * wave + 4 * kg) * GN + (int)(dst >> 2) * 1024 + pi * 16 + l15) * 4u;
    float v[2][8];
    auto load_w = [&](int it, float (&d)[8]) {
#pragma unroll
      for (int j = 0; j < 8; ++j)
        d[j] = __builtin_bit_cast(float, __builtin_amdgcn_raw_buffer_load_b32(r_w, wo + (it - 1) * 1024, (16 * (j >> 2) + (j & 3)) * GN * 4, 0));
    };
    auto max_item = [&](int it, const float (&d)[8]) { max_only(it, d); lds_arrive(&sctl[C_ITEM + it], lane); };
    load_x(row_f, v[0]);
    const unsigned call_v = __hip_atomic_load(ctr, __ATOMIC_RELAXED, __HIP_MEMORY_SCOPE_AGENT);
    COMPILER_FENCE();
    load_w(1, v[1]);
    if (wave == 0) { sctl[lane] = 0; sctl[64 + lane] = 0; WAIT_LGKM0(); }
    __builtin_amdgcn_s_barrier();
    const __amdgpu_buffer_rsrc_t r_d = mk_rsrc(a.ws[dst], WS_USED);
    max_item(0, v[0]);
    set_call(call_v);
    const unsigned xa_d = XA_OFF + par * XA_PAR + (unsigned)rank * XA_SRC + (unsigned)(wave * 32 + pi) * 1024u + l16;
    const unsigned wb_d = WB_OFF + par * WB_PAR + (unsigned)rank * WB_SRC + (unsigned)(wave * 64 + pi) * 1024u + l16;
    const unsigned ex_d = EX_OFF + (par * 8 + rank) * 2048u + pi * 64u, ew_d = EW_OFF + (par * 8 + rank) * 4096u + pi * 64u;
    if (wave >= 4) max_item(1, v[1]);
    unsigned grid = 0;
#pragma unroll
    for (int it = 0; it < 5; ++it) {
      lds_wait(&sctl[C_ITEM + it], NW);
      const int e = exp_of(__hip_atomic_load(&sctl[16 * it + l15], __ATOMIC_RELAXED, __HIP_MEMORY_SCOPE_WORKGROUP));
      u4 f = pack8(v[it & 1], -e, tag);
      asm volatile("" : "+v"(f));
      if (it == 0) {
        if (wave == 0) {
          const unsigned now = now256();
          const unsigned phs = ((now & 0x3fffffffu) + 16u * SLOT256 - (unsigned)pi * SLOT256) % (16u * SLOT256);
          grid = (now - phs + 16u * SLOT256) | 1u;
          if (lane == 0) __hip_atomic_store(&sctl[C_GRID], grid, __ATOMIC_RELAXED, __HIP_MEMORY_SCOPE_WORKGROUP);
        } else {
          lds_wait(&sctl[C_GRID], 1u);
          grid = __builtin_amdgcn_readfirstlane(__hip_atomic_load(&sctl[C_GRID], __ATOMIC_RELAXED, __HIP_MEMORY_SCOPE_WORKGROUP));
        }
        STAMP(1);
      }
      wait_until(grid + (unsigned)(it * 8 + wave) * (16u * SLOT256));
      st128(r_d, it == 0 ? xa_d : wb_d + (it - 1) * 16384u, f);
      if (wave == 1 && lane < 16) st32(r_d, lane * 4, it == 0 ? ex_d : ew_d + (it - 1) * 1024u, cw | (unsigned)(e + 512));
      if (it == 0) {
        if (wave < 4) max_item(1, v[1]);
        if (wave == 0 && lane == 0 && pi == TE_PI)
          __hip_atomic_store((unsigned long long*)(me + TE_OFF + dst * 128u),
                             ((unsigned long long)call << 32) | (grid + 40u * 16u * SLOT256), __ATOMIC_RELAXED, __HIP_MEMORY_SCOPE_AGENT);
        if (cross) {
          const unsigned mark = mark_of(lane);
          st128(r_out, mark, u4{EMPTY, EMPTY, EMPTY, EMPTY});
          st128(r_out, mark + 2048u, u4{EMPTY, EMPTY, EMPTY, EMPTY});
        }
      }
      if (it == 1 && wave == 0 && lane == 0) __hip_atomic_store(ctr, call, __ATOMIC_RELAXED, __HIP_MEMORY_SCOPE_AGENT);
      if (it + 2 < 5) { load_w(it + 2, v[it & 1]); max_item(it + 2, v[it & 1]); }
      if (it == 0) WAIT_VM0();
    }
    wait_until(grid + 39u * 16u * SLOT256 + 2048u);
    STAMP(2);
  } else {
    const unsigned call_v = __hip_atomic_load(ctr, __ATOMIC_RELAXED, __HIP_MEMORY_SCOPE_AGENT);
    if (wave == 0) { sctl[lane] = 0; sctl[64 + lane] = 0; WAIT_LGKM0(); }
    set_call(call_v);
    WAIT_VM0();
    __builtin_amdgcn_s_barrier();
    if (wave == 0 && lane == 0) __hip_atomic_store(ctr, call, __ATOMIC_RELAXED, __HIP_MEMORY_SCOPE_AGENT);
    if (late) {
      __builtin_amdgcn_s_sleep(32);
      float v[4][8];
#pragma unroll
      for (int u = 0; u < 4; ++u) load_x(256 * (int)dst + 16 * ((pi >> 2) + 4 * u), v[u]);
#pragma unroll
      for (int u = 0; u < 4; ++u) max_only(u, v[u]);
      WAIT_LGKM0();
      __builtin_amdgcn_s_barrier();
      COMPILER_FENCE();
#pragma unroll
      for (int u = 0; u < 4; ++u) {
        const int e = exp_of(__hip_atomic_load(&sctl[16 * u + l15], __ATOMIC_RELAXED, __HIP_MEMORY_SCOPE_WORKGROUP));
        *(u4*)(smem + LDS_XO + (wave * 4 + u) * 1024 + l16) = pack8(v[u], -e, tag);
      }
    }
  }

  Waiter wt; wt.tmo = (unsigned*)(me + TMO_OFF); wt.call = call; wt.start();
  const unsigned wb_base = WB_OFF + par * WB_PAR + (unsigned)src * WB_SRC + (unsigned)(panel * 4) * 1024u;
  const unsigned xa_base = XA_OFF + par * XA_PAR + (unsigned)src * XA_SRC + (unsigned)(rb * 2) * 1024u;
  const unsigned tagm = tag ? 0u : ~0u;
  int* sc = (int*)(smem + LDS_SC + wave * 512);
  const unsigned hoff = (unsigned)((src + (lane & 3) * 2) & 7) * 128u + 112u;

  u4 A[8][2], B[8][4];
  auto get_A = [&]() {
    for (;;) {
      const unsigned pv = ld32(r_me, (lane & 1) * 1024u + hoff, xa_base + 7 * 32768u);
      if (__all((pv ^ tagm) & 1u) || wt.expired()) break;
    }
    COMPILER_FENCE();
    for (;;) {
      unsigned ok = 1u;
#pragma unroll
      for (int kb = 0; kb < 8; ++kb)
#pragma unroll
        for (int mi = 0; mi < 2; ++mi) A[kb][mi] = ld128(r_me, l16 + mi * 1024u, xa_base + kb * 32768u);
      const unsigned exv = ld32(r_me, (lane & 31) * 4, EX_OFF + (par * 8 + src) * 2048u + rb * 128u);
#pragma unroll
      for (int kb = 0; kb < 8; ++kb)
#pragma unroll
        for (int mi = 0; mi < 2; ++mi) ok &= A[kb][mi].x ^ tagm;
      sc[64 + lane] = (int)(exv & 1023u) - 1024;
      if (__all((ok & 1u) && (exv ^ cw) < 1024u) || wt.expired()) break;
    }
  };
  auto get_B = [&]() {
    for (;;) {
      const unsigned pv = ld32(r_me, (lane & 3) * 1024u + hoff, wb_base + 7 * 65536u);
      if (__all((pv ^ tagm) & 1u) || wt.expired()) break;
    }
    COMPILER_FENCE();
    for (;;) {
      unsigned ok = 1u;
#pragma unroll
      for (int kb = 0; kb < 8; ++kb)
#pragma unroll
        for (int ni = 0; ni < 4; ++ni) B[kb][ni] = ld128(r_me, l16 + ni * 1024u, wb_base + kb * 65536u);
      const unsigned ewv = ld32(r_me, lane * 4, EW_OFF + (par * 8 + src) * 4096u + panel * 256u);
#pragma unroll
      for (int kb = 0; kb < 8; ++kb)
#pragma unroll
        for (int ni = 0; ni < 4; ++ni) ok &= B[kb][ni].x ^ tagm;
      sc[lane] = (int)(ewv & 1023u);
      if (__all((ok & 1u) && (ewv ^ cw) < 1024u) || wt.expired()) break;
    }
  };
  if (foreign) { get_A(); STAMP(1); get_B(); } else { get_B(); STAMP(3); get_A(); }
  STAMP(4);

  f4 acc[2][4];
#pragma unroll
  for (int mi = 0; mi < 2; ++mi)
#pragma unroll
    for (int ni = 0; ni < 4; ++ni) acc[mi][ni] = f4{0.f, 0.f, 0.f, 0.f};
#pragma unroll
  for (int kb = 0; kb < 8; ++kb)
#pragma unroll
    for (int mi = 0; mi < 2; ++mi)
#pragma unroll
      for (int ni = 0; ni < 4; ++ni)
        acc[mi][ni] = __builtin_amdgcn_mfma_f32_16x16x32_f16(__builtin_bit_cast(h8, B[kb][ni]), __builtin_bit_cast(h8, A[kb][mi]), acc[mi][ni], 0, 0, 0);

  int ln = lane;
  asm volatile("" : "+v"(ln));
  const int e15 = ln & 15, ekg = ln >> 4;
  {
    WAIT_LGKM0();
    int exv[2];
    i4 ewv[4];
#pragma unroll
    for (int mi = 0; mi < 2; ++mi) exv[mi] = sc[64 + mi * 16 + e15];
#pragma unroll
    for (int ni = 0; ni < 4; ++ni) ewv[ni] = *(const i4*)(sc + ni * 16 + ekg * 4);
#pragma unroll
    for (int mi = 0; mi < 2; ++mi)
#pragma unroll
      for (int ni = 0; ni < 4; ++ni) {
        f4 o;
        o.x = __builtin_amdgcn_ldexpf(acc[mi][ni].x, exv[mi] + ewv[ni].x);
        o.y = __builtin_amdgcn_ldexpf(acc[mi][ni].y, exv[mi] + ewv[ni].y);
        o.z = __builtin_amdgcn_ldexpf(acc[mi][ni].z, exv[mi] + ewv[ni].z);
        o.w = __builtin_amdgcn_ldexpf(acc[mi][ni].w, exv[mi] + ewv[ni].w);
        const int q = ni * 4 + ekg;
        *(f4*)(smem + LDS_PART + wave * 8192 + ((mi * 4 + ni) * 64 + (ln & 48) + (e15 ^ q)) * 16) = o;
      }
  }
  if (wt.gave_up && lane == 0) __hip_atomic_store(&sctl[C_DEAD], 1u, __ATOMIC_RELAXED, __HIP_MEMORY_SCOPE_WORKGROUP);
  WAIT_LGKM0();
  __builtin_amdgcn_s_barrier();
  COMPILER_FENCE();
  const bool dead = __hip_atomic_load(&sctl[C_DEAD], __ATOMIC_RELAXED, __HIP_MEMORY_SCOPE_WORKGROUP) != 0;
  const int m = wave * 4 + ekg, q = e15;
  const int idx = ((m >> 4) * 4 + (q >> 2)) * 64 + (q & 3) * 16 + ((m & 15) ^ q);
  f4 s = *(const f4*)(smem + LDS_PART + idx * 16);
#pragma unroll
  for (int w = 1; w < NW; ++w) s += *(const f4*)(smem + LDS_PART + w * 8192 + idx * 16);
  u4 o;
#pragma unroll
  for (int e = 0; e < 4; ++e) {
    const float y = s[e];
    const float c = __builtin_fminf(__builtin_fmaxf(y, -60.f), 60.f);
    const unsigned r = __float_as_uint(y * __builtin_amdgcn_rcpf(1.f + __expf(-c)));
    o[e] = (dead || r == EMPTY) ? QNAN : r;
  }
  if (foreign) {
    const unsigned owner = (unsigned)((((rank >> 2) ^ 1) << 2) | (rb >> 1));
    const unsigned row = (unsigned)(64 * (rank & 3) + 32 * (rb & 1) + m);
    const unsigned col = (unsigned)((rank >> 2) * 1024 + panel * 64 + q * 4);
    const __amdgpu_buffer_rsrc_t r_ow = mk_rsrc(a.out[owner & 7], OUT_BYTES);
    asm volatile("" : "+v"(o));
    STAMP(2);
    auto stream_time = [&](unsigned d, bool& known) {
      const unsigned long long te = __hip_atomic_load((unsigned long long*)(me + TE_OFF + d * 128u), __ATOMIC_RELAXED, __HIP_MEMORY_SCOPE_AGENT);
      known = __builtin_amdgcn_readfirstlane((unsigned)(te >> 32)) == call;
      return (unsigned)__builtin_amdgcn_readfirstlane((unsigned)te);
    };
    bool known_o, known_d;
    const unsigned te_o = stream_time(owner, known_o);
    if (late) {
      const unsigned te_d = stream_time(dst, known_d);
      const __amdgpu_buffer_rsrc_t r_d = mk_rsrc(a.ws[dst], WS_USED);
      const unsigned xa_d = XA_OFF + par * XA_PAR + (unsigned)rank * XA_SRC + (unsigned)(wave * 32 + 16 + (pi >> 2)) * 1024u + ln * 16u;
      u4 f[4];
#pragma unroll
      for (int u = 0; u < 4; ++u) f[u] = *(const u4*)(smem + LDS_XO + (wave * 4 + u) * 1024 + ln * 16);
      const unsigned ew = cw | (unsigned)(exp_of(sctl[ln]) + 512);
#pragma unroll
      for (int u = 0; u < 4; ++u) {
        if (known_d) wait_until(te_d + (unsigned)(wave * 16 + (pi >> 2) + 4 * u) * SLOT256);
        st128(r_d, xa_d + u * 4096u, f[u]);
      }
      if (wave == 1) st32(r_d, e15 * 4 + ekg * 256, EX_OFF + (par * 8 + rank) * 2048u + 1024u + (pi >> 2) * 64u, ew);
    }
    if (known_o) wait_until(te_o + 128u * SLOT256 + OGAP256 + (unsigned)((panel * 2 + (rb & 1)) * 8 + wave) * OSLOT256);
    STAMP(3);
    st128(r_ow, (row * GN + col) * 4u, o);
  } else {
    const unsigned oo = (unsigned)(((32 * (rb - 8) + m) * GN + (rank >> 2) * 1024 + panel * 64 + q * 4) * 4);
    __builtin_amdgcn_raw_buffer_store_b128(o, r_out, (int)oo, 0, OWN_AUX);
    STAMP(5);
    if (cross) {
      const unsigned mark = mark_of(ln);
      for (;;) {
        const u4 t = ld128(r_out, mark, 0), u = ld128(r_out, mark + 2048u, 0);
        if (__all(t.x != EMPTY && t.y != EMPTY && t.z != EMPTY && t.w != EMPTY && u.x != EMPTY && u.y != EMPTY && u.z != EMPTY && u.w != EMPTY)) break;
        if (wt.expired_fast()) {
          WAIT_VM0();
          __builtin_amdgcn_raw_buffer_store_b32(QNAN, r_out, (int)oo, 0, OWN_AUX);
          break;
        }
      }
    }
    STAMP(6);
  }
}

extern "C" void kernel_launch(int rank, int nranks, ncclComm_t comm, hipStream_t stream, void* const* d_in,
                              const int* in_sizes, int n_in, void* d_out, int out_size, const int* in_dtypes,
                              int out_dtype, void* const* d_out_all, void* const* d_ws_all, size_t ws_bytes) {
  if (nranks != NR || (unsigned)rank >= (unsigned)NR || ws_bytes < WS_USED || n_in < 2 || in_sizes[0] != 2048 * KS ||
      in_sizes[1] != KS * GN || out_size != 256 * GN) {
    fprintf(stderr, "unexpected problem shape\n");
    return;
  }
  Args a;
  a.x = (const float*)d_in[0];
  a.w = (const float*)d_in[1];
  for (int p = 0; p < NR; ++p) { a.ws[p] = (char*)d_ws_all[p]; a.out[p] = (float*)d_out_all[p]; }
  a.out[rank] = (float*)d_out;
  a.me = a.ws[rank];
  a.rank = rank;
  fused_gemm_rs_silu<<<NWG, NTHREADS, 0, stream>>>(a);
}
```

```cpp
#include <hip/hip_runtime.h>
#include <rccl/rccl.h>
#include <cstdio>

typedef _Float16 h8 __attribute__((ext_vector_type(8)));
typedef float f4 __attribute__((ext_vector_type(4)));
typedef unsigned u4 __attribute__((ext_vector_type(4)));
typedef int i4 __attribute__((ext_vector_type(4)));
typedef __attribute__((address_space(1))) unsigned gu32;
typedef __attribute__((address_space(1))) unsigned long long gu64;
typedef __attribute__((address_space(1))) char gchar;

constexpr int NR = 8;
constexpr int GN = 2048;
constexpr int KS = 256;
constexpr int NWG = 256;
constexpr int NW = 8;
constexpr int NTHREADS = NW * 64;

#define SPAD 9
constexpr unsigned WB_SRC = (512u + SPAD) << 10, XA_SRC = (256u + SPAD) << 10;
constexpr unsigned WB_PAR = 8 * WB_SRC;
constexpr unsigned XA_PAR = 8 * XA_SRC;
constexpr unsigned WB_OFF = 0;
constexpr unsigned XA_OFF = WB_OFF + 2 * WB_PAR;
constexpr unsigned EW_OFF = XA_OFF + 2 * XA_PAR;
constexpr unsigned EX_OFF = EW_OFF + 65536;
constexpr unsigned CTR_OFF = EX_OFF + 32768;
constexpr unsigned TE_OFF = CTR_OFF + 32768;
constexpr unsigned TMO_OFF = TE_OFF + 1024;
constexpr unsigned WS_USED = TMO_OFF + 128;
constexpr unsigned OUT_BYTES = 256u * GN * 4u;
constexpr unsigned IN_BYTES = 2048u * KS * 4u;
#define DEADLINE 100000000ull
constexpr unsigned long long DEADLINE_TICKS = DEADLINE;
constexpr int AUX_SYS = 17;
#define POLL_SLEEP 8
#define SLOT256 440
#define OSLOT256 SLOT256
constexpr unsigned EMPTY = 0xffffffffu, QNAN = 0x7fc00000u;
#define FAST_SLEEP 2
#define GRID_AHEAD 0
#define XAUX 16
#define TE_PI 0
#define LAND256 25600
#define MARGIN256 23000
#define AGUESS 3
#define SPEC_FROM 6
#define OWN_AUX 17
#define OGAP256 0

struct Args {
  const float* x;
  const float* w;
  char* me;
  float* out_me;
  int rank;
  char* ws[NR];
  float* out[NR];
};

#define WAIT_VM0() asm volatile("s_waitcnt vmcnt(0)" ::: "memory")
#define WAIT_LGKM0() asm volatile("s_waitcnt lgkmcnt(0)" ::: "memory")
#define COMPILER_FENCE() asm volatile("" ::: "memory")

__device__ __forceinline__ __amdgpu_buffer_rsrc_t mk_rsrc(const void* p, unsigned bytes) {
  return __builtin_amdgcn_make_buffer_rsrc((void*)p, (short)0, (int)bytes, 0x00020000);
}
__device__ __forceinline__ void st128(__amdgpu_buffer_rsrc_t r, unsigned voff, u4 v) {
  __builtin_amdgcn_raw_buffer_store_b128(v, r, (int)voff, 0, AUX_SYS);
  __builtin_amdgcn_sched_barrier(0);
}
__device__ __forceinline__ void st32(__amdgpu_buffer_rsrc_t r, unsigned voff, unsigned soff, unsigned v) {
  __builtin_amdgcn_raw_buffer_store_b32(v, r, (int)voff, (int)soff, AUX_SYS);
}
__device__ __forceinline__ u4 ld128(__amdgpu_buffer_rsrc_t r, unsigned voff, unsigned soff) {
  return __builtin_amdgcn_raw_buffer_load_b128(r, (int)voff, (int)soff, AUX_SYS);
}
__device__ __forceinline__ unsigned ld32(__amdgpu_buffer_rsrc_t r, unsigned voff, unsigned soff) {
  return __builtin_amdgcn_raw_buffer_load_b32(r, (int)voff, (int)soff, AUX_SYS);
}
__device__ __forceinline__ float max3abs(float m, float a, float b) {
  float r;
  asm("v_max3_f32 %0, |%1|, |%2|, %3" : "=v"(r) : "v"(a), "v"(b), "v"(m));
  return r;
}
__device__ __forceinline__ int exp_of(unsigned amax_bits) { return (int)(amax_bits >> 23) - (127 + 14); }
__device__ __forceinline__ u4 pack8(const float (&v)[8], int ne, unsigned tag) {
  h8 h;
  h[0] = (_Float16)0.f;
#pragma unroll
  for (int j = 1; j < 8; ++j) h[j] = (_Float16)__builtin_amdgcn_ldexpf(v[j], ne);
  u4 r = __builtin_bit_cast(u4, h);
  const unsigned t = __builtin_bit_cast(unsigned, __builtin_amdgcn_cvt_pkrtz(__builtin_amdgcn_ldexpf(v[0], ne), 0.f));
  unsigned mag = t & 0x7fffu;
  mag = mag < 0x7ffeu ? mag : 0x7ffeu;
  mag += (mag ^ tag) & 1u;
  r.x |= (t & 0x8000u) | mag;
  return r;
}
__device__ __forceinline__ unsigned now256() { return (unsigned)__builtin_amdgcn_s_memrealtime() << 8; }
__device__ __forceinline__ void wait_until(unsigned t256) {
  int d = (int)(t256 - now256());
  if (d > (1 << 22)) return;
  while (d > 4096) { __builtin_amdgcn_s_sleep(2); d = (int)(t256 - now256()); }
  while (d > 0) d = (int)(t256 - now256());
}

struct Waiter {
  gu32* tmo; unsigned call; unsigned long long t0; bool gave_up;
  __device__ __forceinline__ void start() { t0 = __builtin_amdgcn_s_memrealtime(); gave_up = false; }
  __device__ __forceinline__ bool expired() { return expired_t<POLL_SLEEP>(); }
  __device__ __forceinline__ bool expired_fast() { return expired_t<FAST_SLEEP>(); }
  template <int SLEEP> __device__ __forceinline__ bool expired_t() {
    const unsigned long long dt = __builtin_amdgcn_s_memrealtime() - t0;
    if (dt > 4000) {
      if (dt > DEADLINE_TICKS || __hip_atomic_load(tmo, __ATOMIC_RELAXED, __HIP_MEMORY_SCOPE_AGENT) == call) {
        __hip_atomic_store(tmo, call, __ATOMIC_RELAXED, __HIP_MEMORY_SCOPE_AGENT);
        gave_up = true;
        return true;
      }
    }
    __builtin_amdgcn_s_sleep(SLEEP);
    asm volatile("" ::: "memory");
    return false;
  }
};

#define STAMP(j) do {} while (0)

constexpr int LDS_PART = 0;
constexpr int LDS_SC = NW * 8192;
constexpr int LDS_XO = LDS_SC + 4096;
constexpr int LDS_CTL = LDS_XO + 32768;
constexpr int LDS_BYTES = LDS_CTL + 512;
constexpr int C_ITEM = 96, C_ST = 104, C_DEAD = 105, C_GRID = 106;

__device__ __forceinline__ void lds_arrive(unsigned* c, int lane) {
  WAIT_LGKM0();
  if (lane == 0) __hip_atomic_fetch_add(c, 1u, __ATOMIC_RELAXED, __HIP_MEMORY_SCOPE_WORKGROUP);
}
__device__ __forceinline__ void lds_wait(unsigned* c, unsigned n) {
  while (__hip_atomic_load(c, __ATOMIC_RELAXED, __HIP_MEMORY_SCOPE_WORKGROUP) < n) __builtin_amdgcn_s_sleep(1);
  COMPILER_FENCE();
}

__global__ void __launch_bounds__(NTHREADS) fused_gemm_rs_silu(Args a) {
#define TS(i, dep)
  __shared__ __attribute__((aligned(16))) char smem[LDS_BYTES];
  unsigned* sctl = (unsigned*)(smem + LDS_CTL);
  const int b = blockIdx.x;
  const int lane = threadIdx.x & 63;
  const int wave = __builtin_amdgcn_readfirstlane(threadIdx.x >> 6);
  const int x8 = b & 7, ii = b >> 3, pi = ii & 15;
  const float* p_x = a.x; const float* p_w = a.w; gchar* me = (gchar*)a.me; float* p_out = a.out_me; char* p_dst = a.ws[x8];
  int rank = a.rank;
  asm volatile("" : "+s"(p_x), "+s"(p_w), "+s"(me), "+s"(p_out), "+s"(p_dst), "+s"(rank));
  const int l15 = lane & 15, kg = lane >> 4;
  const unsigned l16 = lane * 16;
  gu32* ctr = (gu32*)(me + CTR_OFF) + b * 32;
  const bool foreign = ii >= 16;
  const int ph = pi & 3;
  const int panel = ph * 4 + (x8 >> 1);
  const int rb = (foreign ? 0 : 8) + 4 * (x8 & 1) + (pi >> 2);
  STAMP(0);
  const int src = wave;
  const __amdgpu_buffer_rsrc_t r_me = mk_rsrc((char*)me, WS_USED);
  const __amdgpu_buffer_rsrc_t r_out = mk_rsrc(p_out, OUT_BYTES);
  const bool cross = !foreign && ((x8 ^ rank) & 4);
  const bool late = foreign && ph == 0;
  auto mark_of = [&](int ln) {
    return (unsigned)(((64 * (x8 & 3) + 16 * (pi & 3) + 2 * wave + ((ln >> 4) & 1)) * GN + (x8 >> 2) * 1024 +
                       ((ln >> 5) * 4 + (pi >> 2)) * 64 + (ln & 15) * 4) * 4);
  };
  unsigned call, par, tag, cw;
  auto set_call = [&](unsigned call_v) {
    call = __builtin_amdgcn_readfirstlane(call_v) + 1u;
    par = call & 1u; tag = ((call + 1u) >> 1) & 1u;
    cw = call << 10;
  };
  const unsigned dst = (unsigned)x8;
  const __amdgpu_buffer_rsrc_t r_x = mk_rsrc(p_x, IN_BYTES);
  const unsigned xo = (unsigned)(l15 * KS + 32 * wave + 4 * kg) * 4u;
  auto load_x = [&](int row, float (&d)[8]) {
    const f4 lo = __builtin_bit_cast(f4, __builtin_amdgcn_raw_buffer_load_b128(r_x, xo, row * KS * 4, XAUX));
    const f4 hi = __builtin_bit_cast(f4, __builtin_amdgcn_raw_buffer_load_b128(r_x, xo + 64, row * KS * 4, XAUX));
    d[0] = lo.x; d[1] = lo.y; d[2] = lo.z; d[3] = lo.w; d[4] = hi.x; d[5] = hi.y; d[6] = hi.z; d[7] = hi.w;
  };
  auto max_only = [&](int it, const float (&d)[8]) {
    float m = max3abs(0.f, d[0], d[1]);
    m = max3abs(m, d[2], d[3]); m = max3abs(m, d[4], d[5]); m = max3abs(m, d[6], d[7]);
    __hip_atomic_fetch_max(&sctl[16 * it + l15], __float_as_uint(m), __ATOMIC_RELAXED, __HIP_MEMORY_SCOPE_WORKGROUP);
  };

  if (__builtin_expect(!foreign, 1)) {
    const __amdgpu_buffer_rsrc_t r_w = mk_rsrc(p_w, IN_BYTES);
    const int row_f = 256 * ((((int)(dst >> 2) ^ 1) << 2) | (pi >> 2)) + 64 * (int)(dst & 3) + 16 * (pi & 3);
    const unsigned wo = (unsigned)((32 * wave + 4 * kg) * GN + (int)(dst >> 2) * 1024 + pi * 16 + l15) * 4u;
    float v[2][8];
    auto load_w = [&](int it, float (&d)[8]) {
#pragma unroll
      for (int j = 0; j < 8; ++j)
        d[j] = __builtin_bit_cast(float, __builtin_amdgcn_raw_buffer_load_b32(r_w, wo + (it - 1) * 1024, (16 * (j >> 2) + (j & 3)) * GN * 4, 0));
    };
    auto max_item = [&](int it, const float (&d)[8]) { max_only(it, d); lds_arrive(&sctl[C_ITEM + it], lane); };
    TS(1, "s"(p_x));
    const unsigned xq = (unsigned)((lane >> 2) * KS + 32 * wave + 4 * (lane & 3)) * 4u;
    const f4 xlo = __builtin_bit_cast(f4, __builtin_amdgcn_raw_buffer_load_b128(r_x, xq, row_f * KS * 4, 0));
    const f4 xhi = __builtin_bit_cast(f4, __builtin_amdgcn_raw_buffer_load_b128(r_x, xq + 64, row_f * KS * 4, 0));
    const unsigned call_v = __hip_atomic_load(ctr, __ATOMIC_RELAXED, __HIP_MEMORY_SCOPE_AGENT);
    __builtin_amdgcn_sched_barrier(0);
    if (wave == 0) { sctl[lane] = 0; sctl[64 + lane] = 0; WAIT_LGKM0(); }
    __builtin_amdgcn_s_barrier();
    const __amdgpu_buffer_rsrc_t r_d = mk_rsrc(p_dst, WS_USED);
    unsigned grid;
    {
      const unsigned now = now256();
      grid = now - ((now & 0x3fffffffu) + 16u * SLOT256 - (unsigned)pi * SLOT256) % (16u * SLOT256);
      asm volatile("" : "+s"(grid));
    }
    __builtin_amdgcn_sched_barrier(0);
    TS(2, "v"(xlo.x + xhi.w));
    {
      float m = max3abs(0.f, xlo.x, xlo.y);
      m = max3abs(m, xlo.z, xlo.w); m = max3abs(m, xhi.x, xhi.y); m = max3abs(m, xhi.z, xhi.w);
      __hip_atomic_fetch_max(&sctl[lane >> 2], __float_as_uint(m), __ATOMIC_RELAXED, __HIP_MEMORY_SCOPE_WORKGROUP);
      char* t = smem + LDS_PART + wave * 8192 + ((lane >> 2) + 17 * (lane & 3)) * 16;
      *(f4*)t = xlo; *(f4*)(t + 2048) = xhi;
    }
    WAIT_LGKM0();
    TS(5, "s"(0));
    __builtin_amdgcn_s_barrier();
    COMPILER_FENCE();
    {
      const char* t = smem + LDS_PART + wave * 8192 + (l15 + 17 * kg) * 16;
      const f4 lo = *(const f4*)t, hi = *(const f4*)(t + 2048);
      v[0][0] = lo.x; v[0][1] = lo.y; v[0][2] = lo.z; v[0][3] = lo.w; v[0][4] = hi.x; v[0][5] = hi.y; v[0][6] = hi.z; v[0][7] = hi.w;
    }
    set_call(call_v);
    const unsigned xa_d = XA_OFF + par * XA_PAR + (unsigned)rank * XA_SRC + (unsigned)(wave * 32 + pi) * 1024u + l16;
    const unsigned wb_d = WB_OFF + par * WB_PAR + (unsigned)rank * WB_SRC + (unsigned)(wave * 64 + pi) * 1024u + l16;
    const unsigned ex_d = EX_OFF + (par * 8 + rank) * 2048u + pi * 64u, ew_d = EW_OFF + (par * 8 + rank) * 4096u + pi * 64u;
#pragma unroll
    for (int it = 0; it < 5; ++it) {
      if (it) lds_wait(&sctl[C_ITEM + it], NW);
      const int e = exp_of(__hip_atomic_load(&sctl[16 * it + l15], __ATOMIC_RELAXED, __HIP_MEMORY_SCOPE_WORKGROUP));
      u4 f = pack8(v[it & 1], -e, tag);
      asm volatile("" : "+v"(f));
      if (it == 0) { TS(3, "v"(f)); }
      if (it == 0) {
        if (wave == 0) {
          unsigned long long t;
          asm volatile("s_memrealtime %0\n\ts_waitcnt lgkmcnt(0)" : "=s"(t) : "v"(f));
          const unsigned now = ((unsigned)t << 8) + GRID_AHEAD;
          while ((int)(now - grid) > 0) grid += 16u * SLOT256;
          grid |= 1u;
          if (lane == 0) __hip_atomic_store(&sctl[C_GRID], grid, __ATOMIC_RELAXED, __HIP_MEMORY_SCOPE_WORKGROUP);
        } else {
          lds_wait(&sctl[C_GRID], 1u);
          grid = __builtin_amdgcn_readfirstlane(__hip_atomic_load(&sctl[C_GRID], __ATOMIC_RELAXED, __HIP_MEMORY_SCOPE_WORKGROUP));
        }
        STAMP(1);
        if (wave >= 4) { load_w(1, v[1]); if (wave >= 6) max_item(1, v[1]); }
      }
      wait_until(grid + (unsigned)(it * 8 + wave) * (16u * SLOT256));
      st128(r_d, it == 0 ? xa_d : wb_d + (it - 1) * 16384u, f);
      if (it == 0) { TS(4, "s"(0)); }
      if (wave == 1 && lane < 16) st32(r_d, lane * 4, it == 0 ? ex_d : ew_d + (it - 1) * 1024u, cw | (unsigned)(e + 512));
      if (it == 0) {
        if (wave < 4) load_w(1, v[1]);
        if (wave == 0 && lane == 0 && pi == TE_PI)
          __hip_atomic_store((gu64*)(me + TE_OFF + dst * 128u),
                             ((unsigned long long)call << 32) | (grid + 40u * 16u * SLOT256), __ATOMIC_RELAXED, __HIP_MEMORY_SCOPE_AGENT);
        if (cross) {
          const unsigned mark = mark_of(lane);
          st128(r_out, mark, u4{EMPTY, EMPTY, EMPTY, EMPTY});
          st128(r_out, mark + 2048u, u4{EMPTY, EMPTY, EMPTY, EMPTY});
        }
      }
      if (it == 1 && wave == 0 && lane == 0) __hip_atomic_store(ctr, call, __ATOMIC_RELAXED, __HIP_MEMORY_SCOPE_AGENT);
      if (it + 2 < 5) {
        load_w(it + 2, v[it & 1]);
        if (it == 0 && wave < 6) max_item(1, v[1]);
        max_item(it + 2, v[it & 1]);
      }
      if (it == 0) WAIT_VM0();
    }
    wait_until(grid + 39u * 16u * SLOT256 + 2048u);
    STAMP(2);
  } else {
    const unsigned call_v = __hip_atomic_load(ctr, __ATOMIC_RELAXED, __HIP_MEMORY_SCOPE_AGENT);
    if (wave == 0) { sctl[lane] = 0; sctl[64 + lane] = 0; WAIT_LGKM0(); }
    set_call(call_v);
    WAIT_VM0();
    __builtin_amdgcn_s_barrier();
    if (wave == 0 && lane == 0) __hip_atomic_store(ctr, call, __ATOMIC_RELAXED, __HIP_MEMORY_SCOPE_AGENT);
    if (late) {
      __builtin_amdgcn_s_sleep(32);
      float v[4][8];
#pragma unroll
      for (int u = 0; u < 4; ++u) load_x(256 * (int)dst + 16 * ((pi >> 2) + 4 * u), v[u]);
#pragma unroll
      for (int u = 0; u < 4; ++u) max_only(u, v[u]);
      WAIT_LGKM0();
      __builtin_amdgcn_s_barrier();
      COMPILER_FENCE();
#pragma unroll
      for (int u = 0; u < 4; ++u) {
        const int e = exp_of(__hip_atomic_load(&sctl[16 * u + l15], __ATOMIC_RELAXED, __HIP_MEMORY_SCOPE_WORKGROUP));
        *(u4*)(smem + LDS_XO + (wave * 4 + u) * 1024 + l16) = pack8(v[u], -e, tag);
      }
    }
  }

  Waiter wt; wt.tmo = (gu32*)(me + TMO_OFF); wt.call = call; wt.start();
  const unsigned wb_base = WB_OFF + par * WB_PAR + (unsigned)src * WB_SRC + (unsigned)(panel * 4) * 1024u;
  const unsigned xa_base = XA_OFF + par * XA_PAR + (unsigned)src * XA_SRC + (unsigned)(rb * 2) * 1024u;
  const unsigned tagm = tag ? 0u : ~0u;
  int* sc = (int*)(smem + LDS_SC + wave * 512);
  const unsigned hoff = (unsigned)((src + (lane & 3) * 2) & 7) * 128u + 112u;

  u4 A[8][2], B[8][4];
  auto get_A = [&]() {
    for (;;) {
      const unsigned pv = ld32(r_me, (lane & 1) * 1024u + hoff, xa_base + 7 * 32768u);
      if (__all((pv ^ tagm) & 1u) || wt.expired()) break;
    }
    COMPILER_FENCE();
    for (;;) {
      unsigned ok = 1u;
#pragma unroll
      for (int kb = 0; kb < 8; ++kb)
#pragma unroll
        for (int mi = 0; mi < 2; ++mi) A[kb][mi] = ld128(r_me, l16 + mi * 1024u, xa_base + kb * 32768u);
      const unsigned exv = ld32(r_me, (lane & 31) * 4, EX_OFF + (par * 8 + src) * 2048u + rb * 128u);
#pragma unroll
      for (int kb = 0; kb < 8; ++kb)
#pragma unroll
        for (int mi = 0; mi < 2; ++mi) ok &= A[kb][mi].x ^ tagm;
      sc[64 + lane] = (int)(exv & 1023u) - 1024;
      if (__all((ok & 1u) && (exv ^ cw) < 1024u) || wt.expired()) break;
    }
  };
  auto get_B = [&]() {
    for (;;) {
      const unsigned pv = ld32(r_me, (lane & 3) * 1024u + hoff, wb_base + 7 * 65536u);
      if (__all((pv ^ tagm) & 1u) || wt.expired()) break;
    }
    COMPILER_FENCE();
    for (;;) {
      unsigned ok = 1u;
#pragma unroll
      for (int kb = 0; kb < 8; ++kb)
#pragma unroll
        for (int ni = 0; ni < 4; ++ni) B[kb][ni] = ld128(r_me, l16 + ni * 1024u, wb_base + kb * 65536u);
      const unsigned ewv = ld32(r_me, lane * 4, EW_OFF + (par * 8 + src) * 4096u + panel * 256u);
#pragma unroll
      for (int kb = 0; kb < 8; ++kb)
#pragma unroll
        for (int ni = 0; ni < 4; ++ni) ok &= B[kb][ni].x ^ tagm;
      sc[lane] = (int)(ewv & 1023u);
      if (__all((ok & 1u) && (ewv ^ cw) < 1024u) || wt.expired()) break;
    }
  };
  if (foreign) { get_A(); STAMP(1); get_B(); } else { get_B(); STAMP(3); get_A(); }
  STAMP(4);

  f4 acc[2][4];
#pragma unroll
  for (int mi = 0; mi < 2; ++mi)
#pragma unroll
    for (int ni = 0; ni < 4; ++ni) acc[mi][ni] = f4{0.f, 0.f, 0.f, 0.f};
#pragma unroll
  for (int kb = 0; kb < 8; ++kb)
#pragma unroll
    for (int mi = 0; mi < 2; ++mi)
#pragma unroll
      for (int ni = 0; ni < 4; ++ni)
        acc[mi][ni] = __builtin_amdgcn_mfma_f32_16x16x32_f16(__builtin_bit_cast(h8, B[kb][ni]), __builtin_bit_cast(h8, A[kb][mi]), acc[mi][ni], 0, 0, 0);

  int ln = lane;
  asm volatile("" : "+v"(ln));
  const int e15 = ln & 15, ekg = ln >> 4;
  {
    WAIT_LGKM0();
    int exv[2];
    i4 ewv[4];
#pragma unroll
    for (int mi = 0; mi < 2; ++mi) exv[mi] = sc[64 + mi * 16 + e15];
#pragma unroll
    for (int ni = 0; ni < 4; ++ni) ewv[ni] = *(const i4*)(sc + ni * 16 + ekg * 4);
#pragma unroll
    for (int mi = 0; mi < 2; ++mi)
#pragma unroll
      for (int ni = 0; ni < 4; ++ni) {
        f4 o;
        o.x = __builtin_amdgcn_ldexpf(acc[mi][ni].x, exv[mi] + ewv[ni].x);
        o.y = __builtin_amdgcn_ldexpf(acc[mi][ni].y, exv[mi] + ewv[ni].y);
        o.z = __builtin_amdgcn_ldexpf(acc[mi][ni].z, exv[mi] + ewv[ni].z);
        o.w = __builtin_amdgcn_ldexpf(acc[mi][ni].w, exv[mi] + ewv[ni].w);
        const int q = ni * 4 + ekg;
        *(f4*)(smem + LDS_PART + wave * 8192 + ((mi * 4 + ni) * 64 + (ln & 48) + (e15 ^ q)) * 16) = o;
      }
  }
  if (wt.gave_up && lane == 0) __hip_atomic_store(&sctl[C_DEAD], 1u, __ATOMIC_RELAXED, __HIP_MEMORY_SCOPE_WORKGROUP);
  WAIT_LGKM0();
  __builtin_amdgcn_s_barrier();
  COMPILER_FENCE();
  const bool dead = __hip_atomic_load(&sctl[C_DEAD], __ATOMIC_RELAXED, __HIP_MEMORY_SCOPE_WORKGROUP) != 0;
  const int m = wave * 4 + ekg, q = e15;
  const int idx = ((m >> 4) * 4 + (q >> 2)) * 64 + (q & 3) * 16 + ((m & 15) ^ q);
  f4 s = *(const f4*)(smem + LDS_PART + idx * 16);
#pragma unroll
  for (int w = 1; w < NW; ++w) s += *(const f4*)(smem + LDS_PART + w * 8192 + idx * 16);
  u4 o;
#pragma unroll
  for (int e = 0; e < 4; ++e) {
    const float y = s[e];
    const float c = __builtin_fminf(__builtin_fmaxf(y, -60.f), 60.f);
    const unsigned r = __float_as_uint(y * __builtin_amdgcn_rcpf(1.f + __expf(-c)));
    o[e] = (dead || r == EMPTY) ? QNAN : r;
  }
  if (foreign) {
    const unsigned owner = (unsigned)((((rank >> 2) ^ 1) << 2) | (rb >> 1));
    const unsigned row = (unsigned)(64 * (rank & 3) + 32 * (rb & 1) + m);
    const unsigned col = (unsigned)((rank >> 2) * 1024 + panel * 64 + q * 4);
    const __amdgpu_buffer_rsrc_t r_ow = mk_rsrc(a.out[owner & 7], OUT_BYTES);
    asm volatile("" : "+v"(o));
    STAMP(2);
    auto stream_time = [&](unsigned d, bool& known) {
      const unsigned long long te = __hip_atomic_load((gu64*)(me + TE_OFF + d * 128u), __ATOMIC_RELAXED, __HIP_MEMORY_SCOPE_AGENT);
      known = __builtin_amdgcn_readfirstlane((unsigned)(te >> 32)) == call;
      return (unsigned)__builtin_amdgcn_readfirstlane((unsigned)te);
    };
    bool known_o, known_d;
    const unsigned te_o = stream_time(owner, known_o);
    if (late) {
      const unsigned te_d = stream_time(dst, known_d);
      const __amdgpu_buffer_rsrc_t r_d = mk_rsrc(p_dst, WS_USED);
      const unsigned xa_d = XA_OFF + par * XA_PAR + (unsigned)rank * XA_SRC + (unsigned)(wave * 32 + 16 + (pi >> 2)) * 1024u + ln * 16u;
      u4 f[4];
#pragma unroll
      for (int u = 0; u < 4; ++u) f[u] = *(const u4*)(smem + LDS_XO + (wave * 4 + u) * 1024 + ln * 16);
      const unsigned ew = cw | (unsigned)(exp_of(sctl[ln]) + 512);
#pragma unroll
      for (int u = 0; u < 4; ++u) {
        if (known_d) wait_until(te_d + (unsigned)(wave * 16 + (pi >> 2) + 4 * u) * SLOT256);
        st128(r_d, xa_d + u * 4096u, f[u]);
      }
      if (wave == 1) st32(r_d, e15 * 4 + ekg * 256, EX_OFF + (par * 8 + rank) * 2048u + 1024u + (pi >> 2) * 64u, ew);
    }
    if (known_o) wait_until(te_o + 128u * SLOT256 + OGAP256 + (unsigned)((panel * 2 + (rb & 1)) * 8 + wave) * OSLOT256);
    STAMP(3);
    st128(r_ow, (row * GN + col) * 4u, o);
  } else {
    const unsigned oo = (unsigned)(((32 * (rb - 8) + m) * GN + (rank >> 2) * 1024 + panel * 64 + q * 4) * 4);
    __builtin_amdgcn_raw_buffer_store_b128(o, r_out, (int)oo, 0, OWN_AUX);
    STAMP(5);
    if (cross) {
      const unsigned mark = mark_of(ln);
      for (;;) {
        const u4 t = ld128(r_out, mark, 0), u = ld128(r_out, mark + 2048u, 0);
        if (__all(t.x != EMPTY && t.y != EMPTY && t.z != EMPTY && t.w != EMPTY && u.x != EMPTY && u.y != EMPTY && u.z != EMPTY && u.w != EMPTY)) break;
        if (wt.expired_fast()) {
          WAIT_VM0();
          __builtin_amdgcn_raw_buffer_store_b32(QNAN, r_out, (int)oo, 0, OWN_AUX);
          break;
        }
      }
    }
    STAMP(6);
  }
}

extern "C" void kernel_launch(int rank, int nranks, ncclComm_t comm, hipStream_t stream, void* const* d_in,
                              const int* in_sizes, int n_in, void* d_out, int out_size, const int* in_dtypes,
                              int out_dtype, void* const* d_out_all, void* const* d_ws_all, size_t ws_bytes) {
  if (nranks != NR || (unsigned)rank >= (unsigned)NR || ws_bytes < WS_USED || n_in < 2 || in_sizes[0] != 2048 * KS ||
      in_sizes[1] != KS * GN || out_size != 256 * GN) {
    fprintf(stderr, "unexpected problem shape\n");
    return;
  }
  Args a;
  a.x = (const float*)d_in[0];
  a.w = (const float*)d_in[1];
  for (int p = 0; p < NR; ++p) { a.ws[p] = (char*)d_ws_all[p]; a.out[p] = (float*)d_out_all[p]; }
  a.out[rank] = (float*)d_out;
  a.me = a.ws[rank];
  a.out_me = (float*)d_out;
  a.rank = rank;
  fused_gemm_rs_silu<<<NWG, NTHREADS, 0, stream>>>(a);
}
```

```cpp
#include <hip/hip_runtime.h>
#include <rccl/rccl.h>
#include <cstdio>

typedef _Float16 h8 __attribute__((ext_vector_type(8)));
typedef float f4 __attribute__((ext_vector_type(4)));
typedef unsigned u4 __attribute__((ext_vector_type(4)));
typedef int i4 __attribute__((ext_vector_type(4)));
typedef __attribute__((address_space(1))) unsigned gu32;
typedef __attribute__((address_space(1))) unsigned long long gu64;
typedef __attribute__((address_space(1))) char gchar;

constexpr int NR = 8;
constexpr int GN = 2048;
constexpr int KS = 256;
constexpr int NWG = 256;
constexpr int NW = 8;
constexpr int NTHREADS = NW * 64;

constexpr unsigned WB_SRC = 521u << 10, XA_SRC = 265u << 10;
constexpr unsigned WB_PAR = 8 * WB_SRC;
constexpr unsigned XA_PAR = 8 * XA_SRC;
constexpr unsigned WB_OFF = 0;
constexpr unsigned XA_OFF = WB_OFF + 2 * WB_PAR;
constexpr unsigned EW_OFF = XA_OFF + 2 * XA_PAR;
constexpr unsigned EX_OFF = EW_OFF + 65536;
constexpr unsigned CTR_OFF = EX_OFF + 32768;
constexpr unsigned TE_OFF = CTR_OFF + 32768;
constexpr unsigned TMO_OFF = TE_OFF + 1024;
constexpr unsigned WS_USED = TMO_OFF + 128;
constexpr unsigned OUT_BYTES = 256u * GN * 4u;
constexpr unsigned IN_BYTES = 2048u * KS * 4u;
#define DEADLINE 100000000ull
constexpr unsigned long long DEADLINE_TICKS = DEADLINE;
constexpr int AUX_SYS = 17;
constexpr unsigned EMPTY = 0xffffffffu, QNAN = 0x7fc00000u;
#define POLL_SLEEP 8
#define FAST_SLEEP 2
#define SLOT256 440
#define RQ 2
#define LABELMAP 1
#define OORDER 1
#define LATEW 1
#define SPLITK 4
constexpr unsigned SHORT_N = 16 - 3 * RQ, SHORT_LEN = 8 * SHORT_N, HOLE = 32 * RQ;
constexpr unsigned X_XO = 512 + SHORT_LEN, X_OUT = X_XO + 128;
constexpr unsigned I_RELAY = 512, I_XO = 640 + HOLE;

struct Args {
  const float* x;
  const float* w;
  char* me;
  float* out_me;
  int rank;
  char* ws_dst[NR];
  float* out[NR];
};
__host__ __device__ inline int dst_of(int rank, int label) {
  const bool cross = label <= 2 || label == 7;
  return ((((rank >> 2) ^ (cross ? 1 : 0)) & 1) << 2) | ((cross ? label : label - 6 + rank) & 3);
}

#define WAIT_VM0() asm volatile("s_waitcnt vmcnt(0)" ::: "memory")
#define WAIT_LGKM0() asm volatile("s_waitcnt lgkmcnt(0)" ::: "memory")
#define COMPILER_FENCE() asm volatile("" ::: "memory")

__device__ __forceinline__ __amdgpu_buffer_rsrc_t mk_rsrc(const void* p, unsigned bytes) {
  return __builtin_amdgcn_make_buffer_rsrc((void*)p, (short)0, (int)bytes, 0x00020000);
}
__device__ __forceinline__ void st128(__amdgpu_buffer_rsrc_t r, unsigned voff, u4 v) {
  __builtin_amdgcn_raw_buffer_store_b128(v, r, (int)voff, 0, AUX_SYS);
  __builtin_amdgcn_sched_barrier(0);
}
__device__ __forceinline__ void st32(__amdgpu_buffer_rsrc_t r, unsigned voff, unsigned soff, unsigned v) {
  __builtin_amdgcn_raw_buffer_store_b32(v, r, (int)voff, (int)soff, AUX_SYS);
}
__device__ __forceinline__ u4 ld128(__amdgpu_buffer_rsrc_t r, unsigned voff, unsigned soff) {
  return __builtin_amdgcn_raw_buffer_load_b128(r, (int)voff, (int)soff, AUX_SYS);
}
__device__ __forceinline__ unsigned ld32(__amdgpu_buffer_rsrc_t r, unsigned voff, unsigned soff) {
  return __builtin_amdgcn_raw_buffer_load_b32(r, (int)voff, (int)soff, AUX_SYS);
}
__device__ __forceinline__ float max3abs(float m, float a, float b) {
  float r;
  asm("v_max3_f32 %0, |%1|, |%2|, %3" : "=v"(r) : "v"(a), "v"(b), "v"(m));
  return r;
}
__device__ __forceinline__ int exp_of(unsigned amax_bits) { return (int)(amax_bits >> 23) - (127 + 14); }
__device__ __forceinline__ u4 pack8(const float (&v)[8], int ne, unsigned tag) {
  h8 h;
  h[0] = (_Float16)0.f;
#pragma unroll
  for (int j = 1; j < 8; ++j) h[j] = (_Float16)__builtin_amdgcn_ldexpf(v[j], ne);
  u4 r = __builtin_bit_cast(u4, h);
  const unsigned t = __builtin_bit_cast(unsigned, __builtin_amdgcn_cvt_pkrtz(__builtin_amdgcn_ldexpf(v[0], ne), 0.f));
  unsigned mag = t & 0x7fffu;
  mag = mag < 0x7ffeu ? mag : 0x7ffeu;
  mag += (mag ^ tag) & 1u;
  r.x |= (t & 0x8000u) | mag;
  return r;
}
__device__ __forceinline__ unsigned now256() { return (unsigned)__builtin_amdgcn_s_memrealtime() << 8; }
__device__ __forceinline__ void wait_until(unsigned t256) {
  int d = (int)(t256 - now256());
  if (d > (1 << 22)) return;
  while (d > 4096) { __builtin_amdgcn_s_sleep(2); d = (int)(t256 - now256()); }
  while (d > 0) d = (int)(t256 - now256());
}

struct Waiter {
  gu32* tmo; unsigned call; unsigned long long t0; bool gave_up;
  __device__ __forceinline__ void start() { t0 = __builtin_amdgcn_s_memrealtime(); gave_up = false; }
  __device__ __forceinline__ bool expired() { return expired_t<POLL_SLEEP>(); }
  __device__ __forceinline__ bool expired_fast() { return expired_t<FAST_SLEEP>(); }
  template <int SLEEP> __device__ __forceinline__ bool expired_t() {
    const unsigned long long dt = __builtin_amdgcn_s_memrealtime() - t0;
    if (dt > 4000) {
      if (dt > DEADLINE_TICKS || __hip_atomic_load(tmo, __ATOMIC_RELAXED, __HIP_MEMORY_SCOPE_AGENT) == call) {
        __hip_atomic_store(tmo, call, __ATOMIC_RELAXED, __HIP_MEMORY_SCOPE_AGENT);
        gave_up = true;
        return true;
      }
    }
    __builtin_amdgcn_s_sleep(SLEEP);
    asm volatile("" ::: "memory");
    return false;
  }
};

#define STAMP(j) do {} while (0)

constexpr int LDS_PART = 0;
constexpr int LDS_SC = NW * 8192;
constexpr int LDS_XO = LDS_SC + 4096;
constexpr int LDS_CTL = LDS_XO + 32768;
constexpr int LDS_BYTES = LDS_CTL + 512;
constexpr int C_ITEM = 96, C_DEAD = 105, C_GRID = 106;

__device__ __forceinline__ void lds_arrive(unsigned* c, int lane) {
  WAIT_LGKM0();
  if (lane == 0) __hip_atomic_fetch_add(c, 1u, __ATOMIC_RELAXED, __HIP_MEMORY_SCOPE_WORKGROUP);
}
__device__ __forceinline__ void lds_wait(unsigned* c, unsigned n) {
  while (__hip_atomic_load(c, __ATOMIC_RELAXED, __HIP_MEMORY_SCOPE_WORKGROUP) < n) __builtin_amdgcn_s_sleep(1);
  COMPILER_FENCE();
}

__global__ void __launch_bounds__(NTHREADS) fused_gemm_rs_silu(Args a) {
  __shared__ __attribute__((aligned(16))) char smem[LDS_BYTES];
  unsigned* sctl = (unsigned*)(smem + LDS_CTL);
  const int b = blockIdx.x;
  const int lane = threadIdx.x & 63;
  const int wave = __builtin_amdgcn_readfirstlane(threadIdx.x >> 6);
  const int x8 = b & 7, ii = b >> 3, pi = ii & 15;
  const float* p_x = a.x; const float* p_w = a.w; gchar* me = (gchar*)a.me; float* p_out = a.out_me; char* p_dst = a.ws_dst[x8];
  int rank = a.rank;
  asm volatile("" : "+s"(p_x), "+s"(p_w), "+s"(me), "+s"(p_out), "+s"(p_dst), "+s"(rank));
  const int l15 = lane & 15, kg = lane >> 4;
  const unsigned l16 = lane * 16;
  gu32* ctr = (gu32*)(me + CTR_OFF) + b * 32;
  const bool foreign = ii >= 16;
  const int ph = pi & 3;
  const int panel = ph * 4 + (x8 >> 1);
  const int rb = (foreign ? 0 : 8) + 4 * (x8 & 1) + (pi >> 2);
  const int dst = dst_of(rank, x8);
  const bool xlink = ((dst ^ rank) & 4) != 0;
  STAMP(0);
  const int src = wave;
  const __amdgpu_buffer_rsrc_t r_me = mk_rsrc((char*)me, WS_USED);
  const __amdgpu_buffer_rsrc_t r_out = mk_rsrc(p_out, OUT_BYTES);
  const __amdgpu_buffer_rsrc_t r_d = mk_rsrc(p_dst, WS_USED);
  const __amdgpu_buffer_rsrc_t r_x = mk_rsrc(p_x, IN_BYTES);
  const bool late_w = LATEW && foreign && ph == 0 && !xlink;
  const bool late = late_w || (foreign && ph == ((LATEW && !xlink) ? 1 : 0));
  auto mark_of = [&](int ln) {
    return (unsigned)(((64 * (dst & 3) + 16 * (pi & 3) + 2 * wave + ((ln >> 4) & 1)) * GN + (dst >> 2) * 1024 +
                       ((ln >> 5) * 4 + (pi >> 2)) * 64 + (ln & 15) * 4) * 4);
  };
  unsigned call, par, tag, cw;
  auto set_call = [&](unsigned call_v) {
    call = __builtin_amdgcn_readfirstlane(call_v) + 1u;
    par = call & 1u; tag = ((call + 1u) >> 1) & 1u;
    cw = call << 10;
  };
  auto stream_time = [&](unsigned d, bool& known) {
    const unsigned long long te = __hip_atomic_load((gu64*)(me + TE_OFF + d * 128u), __ATOMIC_RELAXED, __HIP_MEMORY_SCOPE_AGENT);
    known = __builtin_amdgcn_readfirstlane((unsigned)(te >> 32)) == call;
    return (unsigned)__builtin_amdgcn_readfirstlane((unsigned)te);
  };
  auto max_only = [&](int it, const float (&d)[8]) {
    float m = max3abs(0.f, d[0], d[1]);
    m = max3abs(m, d[2], d[3]); m = max3abs(m, d[4], d[5]); m = max3abs(m, d[6], d[7]);
    __hip_atomic_fetch_max(&sctl[16 * it + l15], __float_as_uint(m), __ATOMIC_RELAXED, __HIP_MEMORY_SCOPE_WORKGROUP);
  };

  if (__builtin_expect(!foreign, 1)) {
    const int row_f = 256 * ((((dst >> 2) ^ 1) << 2) | (pi >> 2)) + 64 * (dst & 3) + 16 * (pi & 3);
    const unsigned xq = (unsigned)((lane >> 2) * KS + 32 * wave + 4 * (lane & 3)) * 4u;
    const f4 xlo = __builtin_bit_cast(f4, __builtin_amdgcn_raw_buffer_load_b128(r_x, xq, row_f * KS * 4, 0));
    const f4 xhi = __builtin_bit_cast(f4, __builtin_amdgcn_raw_buffer_load_b128(r_x, xq + 64, row_f * KS * 4, 0));
    const unsigned call_v = __hip_atomic_load(ctr, __ATOMIC_RELAXED, __HIP_MEMORY_SCOPE_AGENT);
    __builtin_amdgcn_sched_barrier(0);
    if (wave == 0) { sctl[lane] = 0; sctl[64 + lane] = 0; WAIT_LGKM0(); }
    __builtin_amdgcn_s_barrier();
    unsigned grid;
    {
      const unsigned now = now256();
      grid = now - ((now & 0x3fffffffu) + 16u * SLOT256 - (unsigned)pi * SLOT256) % (16u * SLOT256);
      asm volatile("" : "+s"(grid));
    }
    __builtin_amdgcn_sched_barrier(0);
    {
      float m = max3abs(0.f, xlo.x, xlo.y);
      m = max3abs(m, xlo.z, xlo.w); m = max3abs(m, xhi.x, xhi.y); m = max3abs(m, xhi.z, xhi.w);
      __hip_atomic_fetch_max(&sctl[lane >> 2], __float_as_uint(m), __ATOMIC_RELAXED, __HIP_MEMORY_SCOPE_WORKGROUP);
      char* t = smem + LDS_PART + wave * 8192 + ((lane >> 2) + 17 * (lane & 3)) * 16;
      *(f4*)t = xlo; *(f4*)(t + 2048) = xhi;
    }
    WAIT_LGKM0();
    __builtin_amdgcn_s_barrier();
    COMPILER_FENCE();
    float v[2][8];
    {
      const char* t = smem + LDS_PART + wave * 8192 + (l15 + 17 * kg) * 16;
      const f4 lo = *(const f4*)t, hi = *(const f4*)(t + 2048);
      v[0][0] = lo.x; v[0][1] = lo.y; v[0][2] = lo.z; v[0][3] = lo.w; v[0][4] = hi.x; v[0][5] = hi.y; v[0][6] = hi.z; v[0][7] = hi.w;
    }
    set_call(call_v);
    const __amdgpu_buffer_rsrc_t r_w = mk_rsrc(p_w, IN_BYTES);
    const unsigned wo = (unsigned)((32 * wave + 4 * kg) * GN + (dst >> 2) * 1024 + pi * 16 + l15) * 4u;
    const bool xr = RQ > 0 && xlink;
    auto phase_of = [&](int it) { return (unsigned)(xr ? (it == 1 ? 0 : it == 2 ? 3 : it - 2) : it - 1); };
    auto load_w = [&](int it, float (&d)[8]) {
      const unsigned o = wo + phase_of(it) * 1024u;
#pragma unroll
      for (int j = 0; j < 8; ++j)
        d[j] = __builtin_bit_cast(float, __builtin_amdgcn_raw_buffer_load_b32(r_w, o, (16 * (j >> 2) + (j & 3)) * GN * 4, 0));
    };
    auto max_item = [&](int it, const float (&d)[8]) { max_only(it, d); lds_arrive(&sctl[C_ITEM + it], lane); };
    const unsigned xa_d = XA_OFF + par * XA_PAR + (unsigned)rank * XA_SRC + (unsigned)(wave * 32 + pi) * 1024u + l16;
    const unsigned wb_d = WB_OFF + par * WB_PAR + (unsigned)rank * WB_SRC + (unsigned)(wave * 64 + pi) * 1024u + l16;
    const unsigned ex_d = EX_OFF + (par * 8 + rank) * 2048u + pi * 64u, ew_d = EW_OFF + (par * 8 + rank) * 4096u + pi * 64u;
    const int ri = pi - (16 - 4 * RQ);
    const bool send3 = !xr || ri < 0 || ri / (RQ ? RQ : 1) == (dst & 3);
    const int idx3 = ri < 0 ? pi : 16 - 4 * RQ + ri % (RQ ? RQ : 1);
#pragma unroll
    for (int it = 0; it < 5; ++it) {
      if (it == 4 && LATEW && !xlink) break;
      if (it) lds_wait(&sctl[C_ITEM + it], NW);
      const int e = exp_of(__hip_atomic_load(&sctl[16 * it + l15], __ATOMIC_RELAXED, __HIP_MEMORY_SCOPE_WORKGROUP));
      u4 f = pack8(v[it & 1], -e, tag);
      asm volatile("" : "+v"(f));
      if (it == 0) {
        if (wave == 0) {
          unsigned long long t;
          asm volatile("s_memrealtime %0\n\ts_waitcnt lgkmcnt(0)" : "=s"(t) : "v"(f));
          const unsigned now = (unsigned)t << 8;
          while ((int)(now - grid) > 0) grid += 16u * SLOT256;
          grid |= 1u;
          if (lane == 0) __hip_atomic_store(&sctl[C_GRID], grid, __ATOMIC_RELAXED, __HIP_MEMORY_SCOPE_WORKGROUP);
        } else {
          lds_wait(&sctl[C_GRID], 1u);
          grid = __builtin_amdgcn_readfirstlane(__hip_atomic_load(&sctl[C_GRID], __ATOMIC_RELAXED, __HIP_MEMORY_SCOPE_WORKGROUP));
        }
        STAMP(1);
        if (wave >= 4) { load_w(1, v[1]); if (wave >= 6) max_item(1, v[1]); }
      }
      int slot = it * 128 + wave * 16;
      if (xr) { if (it == 2) slot = 256 + wave * (int)SHORT_N + idx3 - pi; else if (it > 2) slot += (int)SHORT_LEN - 128; }
      else if (it == 4) slot += (int)HOLE;
      wait_until(grid + (unsigned)slot * SLOT256);
      if (it != 2 || send3) st128(r_d, it == 0 ? xa_d : wb_d + phase_of(it) * 16384u, f);
      if (wave == 1 && lane < 16) st32(r_d, lane * 4, it == 0 ? ex_d : ew_d + phase_of(it) * 1024u, cw | (unsigned)(e + 512));
      if (it == 0) {
        if (wave < 4) load_w(1, v[1]);
        if (wave == 0 && lane == 0 && pi == 0)
          __hip_atomic_store((gu64*)(me + TE_OFF + dst * 128u), ((unsigned long long)call << 32) | grid, __ATOMIC_RELAXED, __HIP_MEMORY_SCOPE_AGENT);
        if (xlink) {
          const unsigned mark = mark_of(lane);
          st128(r_out, mark, u4{EMPTY, EMPTY, EMPTY, EMPTY});
          st128(r_out, mark + 2048u, u4{EMPTY, EMPTY, EMPTY, EMPTY});
        }
      }
      if (it == 1 && wave == 0 && lane == 0) __hip_atomic_store(ctr, call, __ATOMIC_RELAXED, __HIP_MEMORY_SCOPE_AGENT);
      if (it + 2 < 4 || (it == 2 && (xlink || !LATEW))) {
        load_w(it + 2, v[it & 1]);
        if (it == 0) { if (wave < 6) max_item(1, v[1]); WAIT_VM0(); }
        max_item(it + 2, v[it & 1]);
      }
    }
    wait_until(grid + ((xlink ? X_XO : LATEW ? I_RELAY : I_XO) - 16u) * SLOT256 + 2048u);
    STAMP(2);
  } else {
    const unsigned call_v = __hip_atomic_load(ctr, __ATOMIC_RELAXED, __HIP_MEMORY_SCOPE_AGENT);
    if (wave == 0) { sctl[lane] = 0; sctl[64 + lane] = 0; WAIT_LGKM0(); }
    set_call(call_v);
    WAIT_VM0();
    __builtin_amdgcn_s_barrier();
    if (wave == 0 && lane == 0) __hip_atomic_store(ctr, call, __ATOMIC_RELAXED, __HIP_MEMORY_SCOPE_AGENT);
    if (late) {
      __builtin_amdgcn_s_sleep(32);
      const unsigned xo = (unsigned)(l15 * KS + 32 * wave + 4 * kg) * 4u;
      float v[4][8];
      if (late_w) {
        const __amdgpu_buffer_rsrc_t r_w = mk_rsrc(p_w, IN_BYTES);
        const unsigned wo = (unsigned)((32 * wave + 4 * kg) * GN + (dst >> 2) * 1024 + (48 + (pi >> 2)) * 16 + l15) * 4u;
#pragma unroll
        for (int u = 0; u < 4; ++u)
#pragma unroll
          for (int j = 0; j < 8; ++j)
            v[u][j] = __builtin_bit_cast(float, __builtin_amdgcn_raw_buffer_load_b32(r_w, wo + u * 256, (16 * (j >> 2) + (j & 3)) * GN * 4, 0));
      } else {
#pragma unroll
        for (int u = 0; u < 4; ++u) {
          const int row = 256 * dst + 16 * ((pi >> 2) + 4 * u);
          const f4 lo = __builtin_bit_cast(f4, __builtin_amdgcn_raw_buffer_load_b128(r_x, xo, row * KS * 4, 0));
          const f4 hi = __builtin_bit_cast(f4, __builtin_amdgcn_raw_buffer_load_b128(r_x, xo + 64, row * KS * 4, 0));
          v[u][0] = lo.x; v[u][1] = lo.y; v[u][2] = lo.z; v[u][3] = lo.w; v[u][4] = hi.x; v[u][5] = hi.y; v[u][6] = hi.z; v[u][7] = hi.w;
        }
      }
#pragma unroll
      for (int u = 0; u < 4; ++u) max_only(u, v[u]);
      WAIT_LGKM0();
      __builtin_amdgcn_s_barrier();
      COMPILER_FENCE();
#pragma unroll
      for (int u = 0; u < 4; ++u) {
        const int e = exp_of(__hip_atomic_load(&sctl[16 * u + l15], __ATOMIC_RELAXED, __HIP_MEMORY_SCOPE_WORKGROUP));
        *(u4*)(smem + LDS_XO + (wave * 4 + u) * 1024 + l16) = pack8(v[u], -e, tag);
      }
    }
  }

  Waiter wt; wt.tmo = (gu32*)(me + TMO_OFF); wt.call = call; wt.start();
  const unsigned wb_base = WB_OFF + par * WB_PAR + (unsigned)src * WB_SRC + (unsigned)(panel * 4) * 1024u;
  const unsigned xa_base = XA_OFF + par * XA_PAR + (unsigned)src * XA_SRC + (unsigned)(rb * 2) * 1024u;
  const unsigned tagm = tag ? 0u : ~0u;
  int* sc = (int*)(smem + LDS_SC + wave * 512);
  const unsigned hoff = (unsigned)((src + (lane & 3) * 2) & 7) * 128u + 112u;

  u4 A[8][2], B[8][4];
  f4 acc[2][4];
#pragma unroll
  for (int mi = 0; mi < 2; ++mi)
#pragma unroll
    for (int ni = 0; ni < 4; ++ni) acc[mi][ni] = f4{0.f, 0.f, 0.f, 0.f};
#define PROBE_A(kb) do { for (;;) { const unsigned pv = ld32(r_me, (lane & 1) * 1024u + hoff, xa_base + (kb) * 32768u); \
      if (__all((pv ^ tagm) & 1u) || wt.expired()) break; } COMPILER_FENCE(); } while (0)
#define PROBE_B(kb) do { for (;;) { const unsigned pv = ld32(r_me, (lane & 3) * 1024u + hoff, wb_base + (kb) * 65536u); \
      if (__all((pv ^ tagm) & 1u) || wt.expired()) break; } COMPILER_FENCE(); } while (0)
#define ISSUE_A(k0, k1) _Pragma("unroll") for (int kb = (k0); kb < (k1); ++kb) _Pragma("unroll") for (int mi = 0; mi < 2; ++mi) \
      A[kb][mi] = ld128(r_me, l16 + mi * 1024u, xa_base + kb * 32768u)
#define ISSUE_B(k0, k1) _Pragma("unroll") for (int kb = (k0); kb < (k1); ++kb) _Pragma("unroll") for (int ni = 0; ni < 4; ++ni) \
      B[kb][ni] = ld128(r_me, l16 + ni * 1024u, wb_base + kb * 65536u)
#define CHECK_A(k0, k1) _Pragma("unroll") for (int kb = (k0); kb < (k1); ++kb) _Pragma("unroll") for (int mi = 0; mi < 2; ++mi) ok &= A[kb][mi].x ^ tagm
#define CHECK_B(k0, k1) _Pragma("unroll") for (int kb = (k0); kb < (k1); ++kb) _Pragma("unroll") for (int ni = 0; ni < 4; ++ni) ok &= B[kb][ni].x ^ tagm
#define MMA(k0, k1) _Pragma("unroll") for (int kb = (k0); kb < (k1); ++kb) _Pragma("unroll") for (int mi = 0; mi < 2; ++mi) \
      _Pragma("unroll") for (int ni = 0; ni < 4; ++ni) \
        acc[mi][ni] = __builtin_amdgcn_mfma_f32_16x16x32_f16(__builtin_bit_cast(h8, B[kb][ni]), __builtin_bit_cast(h8, A[kb][mi]), acc[mi][ni], 0, 0, 0)
  auto ld_ex = [&]() { return ld32(r_me, (lane & 31) * 4, EX_OFF + (par * 8 + src) * 2048u + rb * 128u); };
  auto ld_ew = [&]() { return ld32(r_me, lane * 4, EW_OFF + (par * 8 + src) * 4096u + panel * 256u); };

  if (foreign) {
    PROBE_A(7);
    for (;;) {
      unsigned ok = 1u;
      ISSUE_A(0, 8);
      const unsigned exv = ld_ex();
      CHECK_A(0, 8);
      sc[64 + lane] = (int)(exv & 1023u) - 1024;
      if (__all((ok & 1u) && (exv ^ cw) < 1024u) || wt.expired()) break;
    }
    STAMP(1);
    if (ph == 3 && !xlink && dst != rank) {
      const unsigned fo = WB_OFF + par * WB_PAR + (unsigned)((((rank >> 2) ^ 1) << 2) | (pi >> 2)) * WB_SRC +
                          (unsigned)(wave * 64 + 64 - 4 * RQ + (rank & 3) * RQ) * 1024u + l16;
      bool known;
      const unsigned te = stream_time(dst, known);
      if (known) wait_until(te + (256u + SHORT_LEN) * SLOT256);
      u4 f[RQ];
      for (;;) {
        unsigned ok = 1u;
#pragma unroll
        for (int u = 0; u < RQ; ++u) { f[u] = ld128(r_me, fo + u * 1024u, 0); }
#pragma unroll
        for (int u = 0; u < RQ; ++u) ok &= f[u].x ^ tagm;
        if (__all(ok & 1u) || wt.expired()) break;
      }
#pragma unroll
      for (int u = 0; u < RQ; ++u) {
        if (known) wait_until(te + (I_RELAY + (unsigned)(wave * 4 * RQ + u * 4 + (pi >> 2))) * SLOT256);
        st128(r_d, fo + u * 1024u, f[u]);
      }
    }
    PROBE_B(SPLITK - 1);
    for (;;) {
      unsigned ok = 1u;
      ISSUE_B(0, SPLITK);
      CHECK_B(0, SPLITK);
      if (__all(ok & 1u) || wt.expired()) break;
    }
    MMA(0, SPLITK);
    PROBE_B(7);
    for (;;) {
      unsigned ok = 1u;
      ISSUE_B(SPLITK, 8);
      const unsigned ewv = ld_ew();
      CHECK_B(SPLITK, 8);
      sc[lane] = (int)(ewv & 1023u);
      if (__all((ok & 1u) && (ewv ^ cw) < 1024u) || wt.expired()) break;
    }
  } else {
    PROBE_B(7);
    bool a1, a2;
    {
      unsigned ok = 1u;
      ISSUE_B(0, 8);
      const unsigned ewv = ld_ew();
      ISSUE_A(0, 8);
      const unsigned exv = ld_ex();
      CHECK_B(0, 8);
      sc[lane] = (int)(ewv & 1023u);
      bool bok = __all((ok & 1u) && (ewv ^ cw) < 1024u);
      ok = 1u;
      CHECK_A(0, SPLITK);
      a1 = __all(ok & 1u);
      ok = 1u;
      CHECK_A(SPLITK, 8);
      sc[64 + lane] = (int)(exv & 1023u) - 1024;
      a2 = __all((ok & 1u) && (exv ^ cw) < 1024u);
      while (!bok && !wt.expired()) {
        ok = 1u;
        ISSUE_B(0, 8);
        const unsigned ew2 = ld_ew();
        CHECK_B(0, 8);
        sc[lane] = (int)(ew2 & 1023u);
        bok = __all((ok & 1u) && (ew2 ^ cw) < 1024u);
      }
    }
    STAMP(3);
    if (!a1) {
      PROBE_A(SPLITK - 1);
      for (;;) {
        unsigned ok = 1u;
        ISSUE_A(0, SPLITK);
        CHECK_A(0, SPLITK);
        if (__all(ok & 1u) || wt.expired()) break;
      }
    }
    MMA(0, SPLITK);
    if (!a2) {
      PROBE_A(7);
      for (;;) {
        unsigned ok = 1u;
        ISSUE_A(SPLITK, 8);
        const unsigned exv = ld_ex();
        CHECK_A(SPLITK, 8);
        sc[64 + lane] = (int)(exv & 1023u) - 1024;
        if (__all((ok & 1u) && (exv ^ cw) < 1024u) || wt.expired()) break;
      }
    }
  }
  STAMP(4);
  MMA(SPLITK, 8);

  int ln = lane;
  asm volatile("" : "+v"(ln));
  const int e15 = ln & 15, ekg = ln >> 4;
  {
    WAIT_LGKM0();
    int exv[2];
    i4 ewv[4];
#pragma unroll
    for (int mi = 0; mi < 2; ++mi) exv[mi] = sc[64 + mi * 16 + e15];
#pragma unroll
    for (int ni = 0; ni < 4; ++ni) ewv[ni] = *(const i4*)(sc + ni * 16 + ekg * 4);
#pragma unroll
    for (int mi = 0; mi < 2; ++mi)
#pragma unroll
      for (int ni = 0; ni < 4; ++ni) {
        f4 o;
        o.x = __builtin_amdgcn_ldexpf(acc[mi][ni].x, exv[mi] + ewv[ni].x);
        o.y = __builtin_amdgcn_ldexpf(acc[mi][ni].y, exv[mi] + ewv[ni].y);
        o.z = __builtin_amdgcn_ldexpf(acc[mi][ni].z, exv[mi] + ewv[ni].z);
        o.w = __builtin_amdgcn_ldexpf(acc[mi][ni].w, exv[mi] + ewv[ni].w);
        const int q = ni * 4 + ekg;
        *(f4*)(smem + LDS_PART + wave * 8192 + ((mi * 4 + ni) * 64 + (ln & 48) + (e15 ^ q)) * 16) = o;
      }
  }
  if (wt.gave_up && lane == 0) __hip_atomic_store(&sctl[C_DEAD], 1u, __ATOMIC_RELAXED, __HIP_MEMORY_SCOPE_WORKGROUP);
  WAIT_LGKM0();
  __builtin_amdgcn_s_barrier();
  COMPILER_FENCE();
  const bool dead = __hip_atomic_load(&sctl[C_DEAD], __ATOMIC_RELAXED, __HIP_MEMORY_SCOPE_WORKGROUP) != 0;
  const int m = wave * 4 + ekg, q = e15;
  const int idx = ((m >> 4) * 4 + (q >> 2)) * 64 + (q & 3) * 16 + ((m & 15) ^ q);
  f4 s = *(const f4*)(smem + LDS_PART + idx * 16);
#pragma unroll
  for (int w = 1; w < NW; ++w) s += *(const f4*)(smem + LDS_PART + w * 8192 + idx * 16);
  u4 o;
#pragma unroll
  for (int e = 0; e < 4; ++e) {
    const float y = s[e];
    const float c = __builtin_fminf(__builtin_fmaxf(y, -60.f), 60.f);
    const unsigned r = __float_as_uint(y * __builtin_amdgcn_rcpf(1.f + __expf(-c)));
    o[e] = (dead || r == EMPTY) ? QNAN : r;
  }
  if (foreign) {
    const unsigned owner = (unsigned)((((rank >> 2) ^ 1) << 2) | (rb >> 1));
    const unsigned row = (unsigned)(64 * (rank & 3) + 32 * (rb & 1) + m);
    const unsigned col = (unsigned)((rank >> 2) * 1024 + panel * 64 + q * 4);
    const __amdgpu_buffer_rsrc_t r_ow = mk_rsrc(a.out[owner & 7], OUT_BYTES);
    asm volatile("" : "+v"(o));
    STAMP(2);
    bool known_o;
    const int oc = xlink ? 0 : 1, oi = x8 >= (xlink ? 2 : 5) ? 1 : 0;
    const int ok_ = (ph < 3 ? oc * 3 + ph : 6 + oc) * 4 + oi * 2 + (rb & 1);
    const unsigned t_o = stream_time(owner, known_o) + (X_OUT + (unsigned)(ok_ * 8 + wave)) * SLOT256;
    bool o_sent = false;
    if (late) {
      bool known_d;
      const unsigned t_x = stream_time(dst, known_d) + ((xlink ? X_XO : late_w ? I_XO - 128u : I_XO) + (unsigned)(wave * 16 + (pi >> 2))) * SLOT256;
      const unsigned xa_d = ln * 16u + (late_w ? WB_OFF + par * WB_PAR + (unsigned)rank * WB_SRC + (unsigned)(wave * 64 + 48 + (pi >> 2)) * 1024u
                                               : XA_OFF + par * XA_PAR + (unsigned)rank * XA_SRC + (unsigned)(wave * 32 + 16 + (pi >> 2)) * 1024u);
      const unsigned sc_d = (pi >> 2) * 64u + (late_w ? EW_OFF + (par * 8 + rank) * 4096u + 3072u : EX_OFF + (par * 8 + rank) * 2048u + 1024u);
      u4 f[4];
#pragma unroll
      for (int u = 0; u < 4; ++u) f[u] = *(const u4*)(smem + LDS_XO + (wave * 4 + u) * 1024 + ln * 16);
      const unsigned ew = cw | (unsigned)(exp_of(sctl[ln]) + 512);
      if (known_o && known_d && (int)(t_o - t_x) < 0) {
        wait_until(t_o);
        st128(r_ow, (row * GN + col) * 4u, o);
        o_sent = true;
      }
#pragma unroll
      for (int u = 0; u < 4; ++u) {
        if (known_d) wait_until(t_x + (unsigned)(4 * u) * SLOT256);
        st128(r_d, xa_d + u * 4096u, f[u]);
      }
      if (wave == 1) st32(r_d, e15 * 4 + ekg * 256, sc_d, ew);
    }
    if (!o_sent) {
      if (known_o) wait_until(t_o);
      STAMP(3);
      st128(r_ow, (row * GN + col) * 4u, o);
    }
  } else {
    const unsigned oo = (unsigned)(((32 * (rb - 8) + m) * GN + (rank >> 2) * 1024 + panel * 64 + q * 4) * 4);
    st128(r_out, oo, o);
    STAMP(5);
    if (xlink) {
      const unsigned mark = mark_of(ln);
      for (;;) {
        const u4 t = ld128(r_out, mark, 0), u = ld128(r_out, mark + 2048u, 0);
        if (__all(t.x != EMPTY && t.y != EMPTY && t.z != EMPTY && t.w != EMPTY && u.x != EMPTY && u.y != EMPTY && u.z != EMPTY && u.w != EMPTY)) break;
        if (wt.expired_fast()) {
          WAIT_VM0();
          st32(r_out, oo, 0, QNAN);
          break;
        }
      }
    }
    STAMP(6);
  }
}

extern "C" void kernel_launch(int rank, int nranks, ncclComm_t comm, hipStream_t stream, void* const* d_in,
                              const int* in_sizes, int n_in, void* d_out, int out_size, const int* in_dtypes,
                              int out_dtype, void* const* d_out_all, void* const* d_ws_all, size_t ws_bytes) {
  if (nranks != NR || (unsigned)rank >= (unsigned)NR || ws_bytes < WS_USED || n_in < 2 || in_sizes[0] != 2048 * KS ||
      in_sizes[1] != KS * GN || out_size != 256 * GN) {
    fprintf(stderr, "unexpected problem shape\n");
    return;
  }
  Args a;
  a.x = (const float*)d_in[0];
  a.w = (const float*)d_in[1];
  for (int p = 0; p < NR; ++p) { a.ws_dst[p] = (char*)d_ws_all[dst_of(rank, p) & 7]; a.out[p] = (float*)d_out_all[p]; }
  a.out[rank] = (float*)d_out;
  a.me = (char*)d_ws_all[rank];
  a.out_me = (float*)d_out;
  a.rank = rank;
  fused_gemm_rs_silu<<<NWG, NTHREADS, 0, stream>>>(a);
}
```

```cpp
#include <hip/hip_runtime.h>
#include <rccl/rccl.h>
#include <cstdio>

typedef _Float16 h8 __attribute__((ext_vector_type(8)));
typedef float f4 __attribute__((ext_vector_type(4)));
typedef unsigned u4 __attribute__((ext_vector_type(4)));
typedef int i4 __attribute__((ext_vector_type(4)));
typedef __attribute__((address_space(1))) unsigned gu32;
typedef __attribute__((address_space(1))) unsigned long long gu64;
typedef __attribute__((address_space(1))) char gchar;

constexpr int NR = 8;
constexpr int GN = 2048;
constexpr int KS = 256;
constexpr int NWG = 256;
constexpr int NW = 8;
constexpr int NTHREADS = NW * 64;

constexpr unsigned WB_SRC = 521u << 10, XA_SRC = 265u << 10;
constexpr unsigned WB_PAR = 8 * WB_SRC;
constexpr unsigned XA_PAR = 8 * XA_SRC;
constexpr unsigned WB_OFF = 0;
constexpr unsigned XA_OFF = WB_OFF + 2 * WB_PAR;
constexpr unsigned EW_OFF = XA_OFF + 2 * XA_PAR;
constexpr unsigned EX_OFF = EW_OFF + 65536;
constexpr unsigned CTR_OFF = EX_OFF + 32768;
constexpr unsigned TE_OFF = CTR_OFF + 32768;
constexpr unsigned TMO_OFF = TE_OFF + 1024;
constexpr unsigned WS_USED = TMO_OFF + 128;
constexpr unsigned OUT_BYTES = 256u * GN * 4u;
constexpr unsigned IN_BYTES = 2048u * KS * 4u;
#define DEADLINE 100000000ull
constexpr unsigned long long DEADLINE_TICKS = DEADLINE;
constexpr int AUX_SYS = 17;
constexpr unsigned EMPTY = 0xffffffffu, QNAN = 0x7fc00000u;
#define POLL_SLEEP 8
#define FAST_SLEEP 2
#define SLOT256 440
#define RQ 2
#define LABELMAP 1
#define OORDER 1
#define LATEW 1
#define BASE_AHEAD 4
#define XSW 7
#define SPLITA 6
#define OWNPOLL 1
#define XSPLIT 1
#define SPLITK 4
constexpr unsigned SHORT_N = 16 - 3 * RQ, SHORT_LEN = 8 * SHORT_N, HOLE = 32 * RQ;
constexpr unsigned X_XO = 512 + SHORT_LEN, X_OUT = X_XO + 128;
constexpr unsigned I_RELAY = 512, I_XO = 640 + HOLE;

struct Args {
  const float* x;
  const float* w;
  char* me;
  float* out_me;
  int rank;
  const float* xf[NR];
  char* ws_dst[NR];
  float* out[NR];
};
__host__ __device__ inline int dst_of(int rank, int label) {
  const bool cross = label <= 2 || label == 7;
  return ((((rank >> 2) ^ (cross ? 1 : 0)) & 1) << 2) | ((cross ? label : label - 6 + rank) & 3);
}

#define WAIT_VM0() asm volatile("s_waitcnt vmcnt(0)" ::: "memory")
#define WAIT_LGKM0() asm volatile("s_waitcnt lgkmcnt(0)" ::: "memory")
#define COMPILER_FENCE() asm volatile("" ::: "memory")

__device__ __forceinline__ __amdgpu_buffer_rsrc_t mk_rsrc(const void* p, unsigned bytes) {
  return __builtin_amdgcn_make_buffer_rsrc((void*)p, (short)0, (int)bytes, 0x00020000);
}
__device__ __forceinline__ void st128(__amdgpu_buffer_rsrc_t r, unsigned voff, u4 v) {
  __builtin_amdgcn_raw_buffer_store_b128(v, r, (int)voff, 0, AUX_SYS);
  __builtin_amdgcn_sched_barrier(0);
}
__device__ __forceinline__ void st32(__amdgpu_buffer_rsrc_t r, unsigned voff, unsigned soff, unsigned v) {
  __builtin_amdgcn_raw_buffer_store_b32(v, r, (int)voff, (int)soff, AUX_SYS);
}
__device__ __forceinline__ u4 ld128(__amdgpu_buffer_rsrc_t r, unsigned voff, unsigned soff) {
  return __builtin_amdgcn_raw_buffer_load_b128(r, (int)voff, (int)soff, AUX_SYS);
}
__device__ __forceinline__ unsigned ld32(__amdgpu_buffer_rsrc_t r, unsigned voff, unsigned soff) {
  return __builtin_amdgcn_raw_buffer_load_b32(r, (int)voff, (int)soff, AUX_SYS);
}
__device__ __forceinline__ float max3abs(float m, float a, float b) {
  float r;
  asm("v_max3_f32 %0, |%1|, |%2|, %3" : "=v"(r) : "v"(a), "v"(b), "v"(m));
  return r;
}
__device__ __forceinline__ int exp_of(unsigned amax_bits) { return (int)(amax_bits >> 23) - (127 + 14); }
__device__ __forceinline__ u4 pack8(const float (&v)[8], int ne, unsigned tag) {
  h8 h;
  h[0] = (_Float16)0.f;
#pragma unroll
  for (int j = 1; j < 8; ++j) h[j] = (_Float16)__builtin_amdgcn_ldexpf(v[j], ne);
  u4 r = __builtin_bit_cast(u4, h);
  const unsigned t = __builtin_bit_cast(unsigned, __builtin_amdgcn_cvt_pkrtz(__builtin_amdgcn_ldexpf(v[0], ne), 0.f));
  unsigned mag = t & 0x7fffu;
  mag = mag < 0x7ffeu ? mag : 0x7ffeu;
  mag += (mag ^ tag) & 1u;
  r.x |= (t & 0x8000u) | mag;
  return r;
}
__device__ __forceinline__ unsigned now256() { return (unsigned)__builtin_amdgcn_s_memrealtime() << 8; }
__device__ __forceinline__ void wait_until(unsigned t256) {
  int d = (int)(t256 - now256());
  if (d > (1 << 22)) return;
  while (d > 4096) { __builtin_amdgcn_s_sleep(2); d = (int)(t256 - now256()); }
  while (d > 0) d = (int)(t256 - now256());
}

struct Waiter {
  gu32* tmo; unsigned call; unsigned long long t0; bool gave_up;
  __device__ __forceinline__ void start() { t0 = __builtin_amdgcn_s_memrealtime(); gave_up = false; }
  __device__ __forceinline__ bool expired() { return expired_t<POLL_SLEEP>(); }
  __device__ __forceinline__ bool expired_fast() { return expired_t<FAST_SLEEP>(); }
  template <int SLEEP> __device__ __forceinline__ bool expired_t() {
    const unsigned long long dt = __builtin_amdgcn_s_memrealtime() - t0;
    if (dt > 4000) {
      if (dt > DEADLINE_TICKS || __hip_atomic_load(tmo, __ATOMIC_RELAXED, __HIP_MEMORY_SCOPE_AGENT) == call) {
        __hip_atomic_store(tmo, call, __ATOMIC_RELAXED, __HIP_MEMORY_SCOPE_AGENT);
        gave_up = true;
        return true;
      }
    }
    __builtin_amdgcn_s_sleep(SLEEP);
    asm volatile("" ::: "memory");
    return false;
  }
};

#define STAMP(j) do {} while (0)

#define ACK(j) do {} while (0)
constexpr int LDS_PART = 0;
constexpr int LDS_SC = NW * 8192;
constexpr int LDS_XO = LDS_SC + 4096;
constexpr int LDS_CTL = LDS_XO + 32768;
constexpr int LDS_BYTES = LDS_CTL + 512;
constexpr int C_ITEM = 96, C_DEAD = 105, C_GRID = 106;

__device__ __forceinline__ void lds_arrive(unsigned* c, int lane) {
  WAIT_LGKM0();
  if (lane == 0) __hip_atomic_fetch_add(c, 1u, __ATOMIC_RELAXED, __HIP_MEMORY_SCOPE_WORKGROUP);
}
__device__ __forceinline__ void lds_wait(unsigned* c, unsigned n) {
  while (__hip_atomic_load(c, __ATOMIC_RELAXED, __HIP_MEMORY_SCOPE_WORKGROUP) < n) __builtin_amdgcn_s_sleep(1);
  COMPILER_FENCE();
}

__global__ void __launch_bounds__(NTHREADS) fused_gemm_rs_silu(Args a) {
  __shared__ __attribute__((aligned(16))) char smem[LDS_BYTES];
  unsigned* sctl = (unsigned*)(smem + LDS_CTL);
  const int b = blockIdx.x;
  const int lane = threadIdx.x & 63;
  const int wave = __builtin_amdgcn_readfirstlane(threadIdx.x >> 6);
  const int x8 = b & 7, ii = b >> 3, pi = ii & 15;
  const bool foreign = ii >= 16;
  typedef __attribute__((address_space(1))) const f4 gf4;
  gchar* me = (gchar*)a.me;
  gu32* ctr = (gu32*)(me + CTR_OFF) + b * 32;
  f4 xlo = {}, xhi = {};
  if (__builtin_expect(!foreign, 1)) {
    const unsigned xq = (unsigned)((256 * (pi >> 2) + 16 * (pi & 3) + (lane >> 2)) * KS + 32 * (int)(threadIdx.x >> 6) + 4 * (lane & 3)) * 4u;
    gchar* px = (gchar*)a.xf[x8];
    xlo = *(gf4*)(px + xq);
    xhi = *(gf4*)(px + xq + 64);
  }
  const unsigned call_v = __hip_atomic_load(ctr, __ATOMIC_RELAXED, __HIP_MEMORY_SCOPE_AGENT);
  __builtin_amdgcn_sched_barrier(0);
  const float* p_x = a.x; const float* p_w = a.w; float* p_out = a.out_me; char* p_dst = a.ws_dst[x8];
  int rank = a.rank;
  asm volatile("" : "+s"(p_x), "+s"(p_w), "+s"(p_out), "+s"(p_dst), "+s"(rank));
  const int l15 = lane & 15, kg = lane >> 4;
  const unsigned l16 = lane * 16;
  const int ph = pi & 3;
  const int panel = ph * 4 + (x8 >> 1);
  const int rb = (foreign ? 0 : 8) + 4 * (x8 & 1) + (pi >> 2);
  const int dst = dst_of(rank, x8);
  const bool xlink = ((dst ^ rank) & 4) != 0;
  STAMP(0);
  const int src = wave;
  const __amdgpu_buffer_rsrc_t r_me = mk_rsrc((char*)me, WS_USED);
  const __amdgpu_buffer_rsrc_t r_out = mk_rsrc(p_out, OUT_BYTES);
  const __amdgpu_buffer_rsrc_t r_d = mk_rsrc(p_dst, WS_USED);
  const __amdgpu_buffer_rsrc_t r_x = mk_rsrc(p_x, IN_BYTES);
  const bool late_w = LATEW && foreign && ph == 0 && !xlink;
  const bool late_h = XSPLIT && foreign && ph == 1 && xlink;
  const bool late = late_w || late_h || (foreign && ph == ((LATEW && !xlink) ? 1 : 0));
  const int lbl2 = 3 + (x8 & 3);
  const int ldst = late_h ? dst_of(rank, lbl2) : dst;
  auto mark_of = [&](int ln) {
    return (unsigned)(((64 * (dst & 3) + 16 * (pi & 3) + 2 * wave + ((ln >> 4) & 1)) * GN + (dst >> 2) * 1024 +
                       ((ln >> 5) * 4 + (pi >> 2)) * 64 + (ln & 15) * 4) * 4);
  };
  unsigned call, par, tag, cw;
  auto set_call = [&](unsigned call_v) {
    call = __builtin_amdgcn_readfirstlane(call_v) + 1u;
    par = call & 1u; tag = ((call + 1u) >> 1) & 1u;
    cw = call << 10;
  };
  auto stream_time = [&](unsigned d, bool& known) {
    const unsigned long long te = __hip_atomic_load((gu64*)(me + TE_OFF + d * 128u), __ATOMIC_RELAXED, __HIP_MEMORY_SCOPE_AGENT);
    known = __builtin_amdgcn_readfirstlane((unsigned)(te >> 32)) == call;
    return (unsigned)__builtin_amdgcn_readfirstlane((unsigned)te);
  };
  auto max_only = [&](int it, const float (&d)[8]) {
    float m = max3abs(0.f, d[0], d[1]);
    m = max3abs(m, d[2], d[3]); m = max3abs(m, d[4], d[5]); m = max3abs(m, d[6], d[7]);
    __hip_atomic_fetch_max(&sctl[16 * it + l15], __float_as_uint(m), __ATOMIC_RELAXED, __HIP_MEMORY_SCOPE_WORKGROUP);
  };

  if (__builtin_expect(!foreign, 1)) {
    if (wave == 0) { sctl[lane] = 0; sctl[64 + lane] = 0; WAIT_LGKM0(); }
    __builtin_amdgcn_s_barrier();
    unsigned grid;
    {
      const unsigned now = now256();
      grid = now - ((now & 0x3fffffffu) + 16u * SLOT256 - (unsigned)pi * SLOT256) % (16u * SLOT256);
      asm volatile("" : "+s"(grid));
    }
    __builtin_amdgcn_sched_barrier(0);
    {
      float m = max3abs(0.f, xlo.x, xlo.y);
      m = max3abs(m, xlo.z, xlo.w); m = max3abs(m, xhi.x, xhi.y); m = max3abs(m, xhi.z, xhi.w);
      __hip_atomic_fetch_max(&sctl[lane >> 2], __float_as_uint(m), __ATOMIC_RELAXED, __HIP_MEMORY_SCOPE_WORKGROUP);
      char* t = smem + LDS_PART + wave * 8192 + ((lane >> 2) + 17 * (lane & 3)) * 16;
      *(f4*)t = xlo; *(f4*)(t + 2048) = xhi;
    }
    WAIT_LGKM0();
    __builtin_amdgcn_s_barrier();
    COMPILER_FENCE();
    float v[2][8];
    {
      const char* t = smem + LDS_PART + wave * 8192 + (l15 + 17 * kg) * 16;
      const f4 lo = *(const f4*)t, hi = *(const f4*)(t + 2048);
      v[0][0] = lo.x; v[0][1] = lo.y; v[0][2] = lo.z; v[0][3] = lo.w; v[0][4] = hi.x; v[0][5] = hi.y; v[0][6] = hi.z; v[0][7] = hi.w;
    }
    set_call(call_v);
    const __amdgpu_buffer_rsrc_t r_w = mk_rsrc(p_w, IN_BYTES);
    const unsigned wo = (unsigned)((32 * wave + 4 * kg) * GN + (dst >> 2) * 1024 + pi * 16 + l15) * 4u;
    const bool xr = RQ > 0 && xlink;
    auto phase_of = [&](int it) { return (unsigned)(xr ? (it == 1 ? 0 : it == 2 ? 3 : it - 2) : it - 1); };
    auto load_w = [&](int it, float (&d)[8]) {
      const unsigned o = wo + phase_of(it) * 1024u;
#pragma unroll
      for (int j = 0; j < 8; ++j)
        d[j] = __builtin_bit_cast(float, __builtin_amdgcn_raw_buffer_load_b32(r_w, o, (16 * (j >> 2) + (j & 3)) * GN * 4, 0));
    };
    auto max_item = [&](int it, const float (&d)[8]) { max_only(it, d); lds_arrive(&sctl[C_ITEM + it], lane); };
    const unsigned xa_d = XA_OFF + par * XA_PAR + (unsigned)rank * XA_SRC + (unsigned)(wave * 32 + pi) * 1024u + l16;
    const unsigned wb_d = WB_OFF + par * WB_PAR + (unsigned)rank * WB_SRC + (unsigned)(wave * 64 + pi) * 1024u + l16;
    const unsigned ex_d = EX_OFF + (par * 8 + rank) * 2048u + pi * 64u, ew_d = EW_OFF + (par * 8 + rank) * 4096u + pi * 64u;
    const int ri = pi - (16 - 4 * RQ);
    const bool send3 = !xr || ri < 0 || ri / (RQ ? RQ : 1) == (dst & 3);
    const int idx3 = ri < 0 ? pi : 16 - 4 * RQ + ri % (RQ ? RQ : 1);
#pragma unroll
    for (int it = 0; it < 5; ++it) {
      if (it == 4 && LATEW && !xlink) break;
      if (it) lds_wait(&sctl[C_ITEM + it], NW);
      const int e = exp_of(__hip_atomic_load(&sctl[16 * it + l15], __ATOMIC_RELAXED, __HIP_MEMORY_SCOPE_WORKGROUP));
      u4 f = pack8(v[it & 1], -e, tag);
      asm volatile("" : "+v"(f));
      if (it == 0) {
        if (wave == 0) {
          unsigned long long t;
          asm volatile("s_memrealtime %0\n\ts_waitcnt lgkmcnt(0)" : "=s"(t) : "v"(f));
          const unsigned now = (unsigned)t << 8;
          while ((int)(now - grid) > 0) grid += 16u * SLOT256;
          grid |= 1u;
          if (lane == 0) {
            __hip_atomic_store(&sctl[C_GRID], grid, __ATOMIC_RELAXED, __HIP_MEMORY_SCOPE_WORKGROUP);
            if (pi == 0)
              __hip_atomic_store((gu64*)(me + TE_OFF + dst * 128u), ((unsigned long long)call << 32) | (now + BASE_AHEAD * SLOT256),
                                 __ATOMIC_RELAXED, __HIP_MEMORY_SCOPE_AGENT);
          }
        } else {
          lds_wait(&sctl[C_GRID], 1u);
          grid = __builtin_amdgcn_readfirstlane(__hip_atomic_load(&sctl[C_GRID], __ATOMIC_RELAXED, __HIP_MEMORY_SCOPE_WORKGROUP));
        }
        STAMP(1);
        if (wave >= 4) { load_w(1, v[1]); if (wave >= 6) max_item(1, v[1]); }
      }
      int slot = it * 128 + wave * 16;
      if (xr) { if (it == 2) slot = 256 + wave * (int)SHORT_N + idx3 - pi; else if (it > 2) slot += (int)SHORT_LEN - 128; }
      else if (it == 4) slot += (int)HOLE;
      wait_until(grid + (unsigned)slot * SLOT256);
      if (it != 2 || send3) { st128(r_d, it == 0 ? xa_d : wb_d + phase_of(it) * 16384u, f); if (it || wave < 4) ACK(it + 1); }
      if (wave == 1 && lane < 16) st32(r_d, lane * 4, it == 0 ? ex_d : ew_d + phase_of(it) * 1024u, cw | (unsigned)(e + 512));
      if (it == 0) {
        if (wave < 4) load_w(1, v[1]);
        if (xlink) {
          const unsigned mark = mark_of(lane);
          st128(r_out, mark, u4{EMPTY, EMPTY, EMPTY, EMPTY});
          st128(r_out, mark + 2048u, u4{EMPTY, EMPTY, EMPTY, EMPTY});
        }
      }
      if (it == 1 && wave == 0 && lane == 0) __hip_atomic_store(ctr, call, __ATOMIC_RELAXED, __HIP_MEMORY_SCOPE_AGENT);
      if (it + 2 < 4 || (it == 2 && (xlink || !LATEW))) {
        load_w(it + 2, v[it & 1]);
        if (it == 0) { if (wave < 6) max_item(1, v[1]); WAIT_VM0(); }
        max_item(it + 2, v[it & 1]);
        if (it == 0) {
          bool known;
          const unsigned g0 = stream_time(dst, known);
          if (known) grid = g0 + (unsigned)pi * SLOT256;
        }
      }
    }
    wait_until(grid + ((xlink ? X_XO : LATEW ? I_RELAY : I_XO) - 16u) * SLOT256 + 2048u);
    STAMP(2);
  } else {
    if (wave == 0) { sctl[lane] = 0; sctl[64 + lane] = 0; WAIT_LGKM0(); }
    set_call(call_v);
    WAIT_VM0();
    __builtin_amdgcn_s_barrier();
    if (wave == 0 && lane == 0) __hip_atomic_store(ctr, call, __ATOMIC_RELAXED, __HIP_MEMORY_SCOPE_AGENT);
    if (late) {
      __builtin_amdgcn_s_sleep(32);
      const unsigned xo = (unsigned)(l15 * KS + 32 * wave + 4 * kg) * 4u;
      float v[4][8];
      if (late_w) {
        const __amdgpu_buffer_rsrc_t r_w = mk_rsrc(p_w, IN_BYTES);
        const unsigned wo = (unsigned)((32 * wave + 4 * kg) * GN + (dst >> 2) * 1024 + (48 + (pi >> 2)) * 16 + l15) * 4u;
#pragma unroll
        for (int u = 0; u < 4; ++u)
#pragma unroll
          for (int j = 0; j < 8; ++j)
            v[u][j] = __builtin_bit_cast(float, __builtin_amdgcn_raw_buffer_load_b32(r_w, wo + u * 256, (16 * (j >> 2) + (j & 3)) * GN * 4, 0));
      } else {
#pragma unroll
        for (int u = 0; u < 4; ++u) {
          const int row = 256 * ldst + 16 * ((pi >> 2) + 4 * u);
          const f4 lo = __builtin_bit_cast(f4, __builtin_amdgcn_raw_buffer_load_b128(r_x, xo, row * KS * 4, 0));
          const f4 hi = __builtin_bit_cast(f4, __builtin_amdgcn_raw_buffer_load_b128(r_x, xo + 64, row * KS * 4, 0));
          v[u][0] = lo.x; v[u][1] = lo.y; v[u][2] = lo.z; v[u][3] = lo.w; v[u][4] = hi.x; v[u][5] = hi.y; v[u][6] = hi.z; v[u][7] = hi.w;
        }
      }
#pragma unroll
      for (int u = 0; u < 4; ++u) max_only(u, v[u]);
      WAIT_LGKM0();
      __builtin_amdgcn_s_barrier();
      COMPILER_FENCE();
#pragma unroll
      for (int u = 0; u < 4; ++u) {
        const int e = exp_of(__hip_atomic_load(&sctl[16 * u + l15], __ATOMIC_RELAXED, __HIP_MEMORY_SCOPE_WORKGROUP));
        *(u4*)(smem + LDS_XO + (wave * 4 + u) * 1024 + l16) = pack8(v[u], -e, tag);
      }
    }
  }

  Waiter wt; wt.tmo = (gu32*)(me + TMO_OFF); wt.call = call; wt.start();
  const unsigned wb_base = WB_OFF + par * WB_PAR + (unsigned)src * WB_SRC + (unsigned)(panel * 4) * 1024u;
  const unsigned xa_base = XA_OFF + par * XA_PAR + (unsigned)src * XA_SRC + (unsigned)(rb * 2) * 1024u;
  const unsigned tagm = tag ? 0u : ~0u;
  int* sc = (int*)(smem + LDS_SC + wave * 512);
  const unsigned hoff = (unsigned)((src + (lane & 3) * 2) & 7) * 128u + 112u;

  u4 A[8][2], B[8][4];
  f4 acc[2][4];
#pragma unroll
  for (int mi = 0; mi < 2; ++mi)
#pragma unroll
    for (int ni = 0; ni < 4; ++ni) acc[mi][ni] = f4{0.f, 0.f, 0.f, 0.f};
#define PROBE_A(kb) do { for (;;) { const unsigned pv = ld32(r_me, (lane & 1) * 1024u + hoff, xa_base + (kb) * 32768u); \
      if (__all((pv ^ tagm) & 1u) || wt.expired()) break; } COMPILER_FENCE(); } while (0)
#define PROBE_B(kb) do { for (;;) { const unsigned pv = ld32(r_me, (lane & 3) * 1024u + hoff, wb_base + (kb) * 65536u); \
      if (__all((pv ^ tagm) & 1u) || wt.expired()) break; } COMPILER_FENCE(); } while (0)
#define ISSUE_A(k0, k1) _Pragma("unroll") for (int kb = (k0); kb < (k1); ++kb) _Pragma("unroll") for (int mi = 0; mi < 2; ++mi) \
      A[kb][mi] = ld128(r_me, l16 + mi * 1024u, xa_base + kb * 32768u)
#define ISSUE_B(k0, k1) _Pragma("unroll") for (int kb = (k0); kb < (k1); ++kb) _Pragma("unroll") for (int ni = 0; ni < 4; ++ni) \
      B[kb][ni] = ld128(r_me, l16 + ni * 1024u, wb_base + kb * 65536u)
#define CHECK_A(k0, k1) _Pragma("unroll") for (int kb = (k0); kb < (k1); ++kb) _Pragma("unroll") for (int mi = 0; mi < 2; ++mi) ok &= A[kb][mi].x ^ tagm
#define CHECK_B(k0, k1) _Pragma("unroll") for (int kb = (k0); kb < (k1); ++kb) _Pragma("unroll") for (int ni = 0; ni < 4; ++ni) ok &= B[kb][ni].x ^ tagm
#define MMA(k0, k1) _Pragma("unroll") for (int kb = (k0); kb < (k1); ++kb) _Pragma("unroll") for (int mi = 0; mi < 2; ++mi) \
      _Pragma("unroll") for (int ni = 0; ni < 4; ++ni) \
        acc[mi][ni] = __builtin_amdgcn_mfma_f32_16x16x32_f16(__builtin_bit_cast(h8, B[kb][ni]), __builtin_bit_cast(h8, A[kb][mi]), acc[mi][ni], 0, 0, 0)
  auto ld_ex = [&]() { return ld32(r_me, (lane & 31) * 4, EX_OFF + (par * 8 + src) * 2048u + rb * 128u); };
  auto ld_ew = [&]() { return ld32(r_me, lane * 4, EW_OFF + (par * 8 + src) * 4096u + panel * 256u); };

  if (foreign) {
    PROBE_A(7);
    for (;;) {
      unsigned ok = 1u;
      ISSUE_A(0, 8);
      const unsigned exv = ld_ex();
      CHECK_A(0, 8);
      sc[64 + lane] = (int)(exv & 1023u) - 1024;
      if (__all((ok & 1u) && (exv ^ cw) < 1024u) || wt.expired()) break;
    }
    STAMP(1);
    if (ph == 3 && !xlink && dst != rank) {
      const unsigned fo = WB_OFF + par * WB_PAR + (unsigned)((((rank >> 2) ^ 1) << 2) | (pi >> 2)) * WB_SRC +
                          (unsigned)(wave * 64 + 64 - 4 * RQ + (rank & 3) * RQ) * 1024u + l16;
      bool known;
      const unsigned te = stream_time(dst, known);
      if (known) wait_until(te + (256u + SHORT_LEN) * SLOT256);
      u4 f[RQ];
      for (;;) {
        unsigned ok = 1u;
#pragma unroll
        for (int u = 0; u < RQ; ++u) { f[u] = ld128(r_me, fo + u * 1024u, 0); }
#pragma unroll
        for (int u = 0; u < RQ; ++u) ok &= f[u].x ^ tagm;
        if (__all(ok & 1u) || wt.expired()) break;
      }
#pragma unroll
      for (int u = 0; u < RQ; ++u) {
        if (known) wait_until(te + (I_RELAY + (unsigned)(wave * 4 * RQ + u * 4 + (pi >> 2))) * SLOT256);
        st128(r_d, fo + u * 1024u, f[u]);
      }
      ACK(1);
    }
    PROBE_B(SPLITK - 1);
    for (;;) {
      unsigned ok = 1u;
      ISSUE_B(0, SPLITK);
      CHECK_B(0, SPLITK);
      if (__all(ok & 1u) || wt.expired()) break;
    }
    MMA(0, SPLITK);
    PROBE_B(7);
    for (;;) {
      unsigned ok = 1u;
      ISSUE_B(SPLITK, 8);
      const unsigned ewv = ld_ew();
      CHECK_B(SPLITK, 8);
      sc[lane] = (int)(ewv & 1023u);
      if (__all((ok & 1u) && (ewv ^ cw) < 1024u) || wt.expired()) break;
    }
    STAMP(4);
    MMA(SPLITK, 8);
  } else {
    PROBE_B(7);
    bool a1, a2;
    {
      unsigned ok = 1u;
      ISSUE_B(0, 8);
      const unsigned ewv = ld_ew();
      ISSUE_A(0, 8);
      const unsigned exv = ld_ex();
      CHECK_B(0, 8);
      sc[lane] = (int)(ewv & 1023u);
      bool bok = __all((ok & 1u) && (ewv ^ cw) < 1024u);
      ok = 1u;
      CHECK_A(0, SPLITA);
      a1 = __all(ok & 1u);
      ok = 1u;
      CHECK_A(SPLITA, 8);
      sc[64 + lane] = (int)(exv & 1023u) - 1024;
      a2 = __all((ok & 1u) && (exv ^ cw) < 1024u);
      while (!bok && !wt.expired()) {
        ok = 1u;
        ISSUE_B(0, 8);
        const unsigned ew2 = ld_ew();
        CHECK_B(0, 8);
        sc[lane] = (int)(ew2 & 1023u);
        bok = __all((ok & 1u) && (ew2 ^ cw) < 1024u);
      }
    }
    STAMP(3);
    if (!a1) {
      PROBE_A(SPLITA - 1);
      for (;;) {
        unsigned ok = 1u;
        ISSUE_A(0, SPLITA);
        CHECK_A(0, SPLITA);
        if (__all(ok & 1u) || wt.expired()) break;
      }
    }
    MMA(0, SPLITA);
    if (!a2) {
      if (!OWNPOLL) PROBE_A(7);
      for (;;) {
        unsigned ok = 1u;
        ISSUE_A(SPLITA, 8);
        const unsigned exv = ld_ex();
        CHECK_A(SPLITA, 8);
        sc[64 + lane] = (int)(exv & 1023u) - 1024;
        if (__all((ok & 1u) && (exv ^ cw) < 1024u) || wt.expired_fast()) break;
      }
    }
    STAMP(4);
    MMA(SPLITA, 8);
  }

  int ln = lane;
  asm volatile("" : "+v"(ln));
  const int e15 = ln & 15, ekg = ln >> 4;
  {
    WAIT_LGKM0();
    int exv[2];
    i4 ewv[4];
#pragma unroll
    for (int mi = 0; mi < 2; ++mi) exv[mi] = sc[64 + mi * 16 + e15];
#pragma unroll
    for (int ni = 0; ni < 4; ++ni) ewv[ni] = *(const i4*)(sc + ni * 16 + ekg * 4);
#pragma unroll
    for (int mi = 0; mi < 2; ++mi)
#pragma unroll
      for (int ni = 0; ni < 4; ++ni) {
        f4 o;
        o.x = __builtin_amdgcn_ldexpf(acc[mi][ni].x, exv[mi] + ewv[ni].x);
        o.y = __builtin_amdgcn_ldexpf(acc[mi][ni].y, exv[mi] + ewv[ni].y);
        o.z = __builtin_amdgcn_ldexpf(acc[mi][ni].z, exv[mi] + ewv[ni].z);
        o.w = __builtin_amdgcn_ldexpf(acc[mi][ni].w, exv[mi] + ewv[ni].w);
        const int q = ni * 4 + ekg;
        *(f4*)(smem + LDS_PART + wave * 8192 + ((mi * 4 + ni) * 64 + (ln & 48) + (e15 ^ q)) * 16) = o;
      }
  }
  if (wt.gave_up && lane == 0) __hip_atomic_store(&sctl[C_DEAD], 1u, __ATOMIC_RELAXED, __HIP_MEMORY_SCOPE_WORKGROUP);
  WAIT_LGKM0();
  __builtin_amdgcn_s_barrier();
  COMPILER_FENCE();
  const bool dead = __hip_atomic_load(&sctl[C_DEAD], __ATOMIC_RELAXED, __HIP_MEMORY_SCOPE_WORKGROUP) != 0;
  const int m = wave * 4 + ekg, q = e15;
  const int idx = ((m >> 4) * 4 + (q >> 2)) * 64 + (q & 3) * 16 + ((m & 15) ^ q);
  f4 s = *(const f4*)(smem + LDS_PART + idx * 16);
#pragma unroll
  for (int w = 1; w < NW; ++w) s += *(const f4*)(smem + LDS_PART + w * 8192 + idx * 16);
  u4 o;
#pragma unroll
  for (int e = 0; e < 4; ++e) {
    const float y = s[e];
    const float c = __builtin_fminf(__builtin_fmaxf(y, -60.f), 60.f);
    const unsigned r = __float_as_uint(y * __builtin_amdgcn_rcpf(1.f + __expf(-c)));
    o[e] = (dead || r == EMPTY) ? QNAN : r;
  }
  if (foreign) {
    const unsigned owner = (unsigned)((((rank >> 2) ^ 1) << 2) | (rb >> 1));
    const unsigned row = (unsigned)(64 * (rank & 3) + 32 * (rb & 1) + m);
    const unsigned col = (unsigned)((rank >> 2) * 1024 + panel * 64 + q * 4);
    const __amdgpu_buffer_rsrc_t r_ow = mk_rsrc(a.out[owner & 7], OUT_BYTES);
    asm volatile("" : "+v"(o));
    STAMP(2);
    bool known_o;
    const int oc = xlink ? 0 : 1, oi = x8 >= (xlink ? 2 : 5) ? 1 : 0;
    const int ok_ = (ph < 3 ? oc * 3 + ph : 6 + oc) * 4 + oi * 2 + (rb & 1);
    const unsigned t_o = stream_time(owner, known_o) + (X_OUT + (unsigned)(ok_ * 8 + wave)) * SLOT256;
    bool o_sent = false;
    if (late && !(XSPLIT && ph == 1 && (wave >= XSW) != late_h)) {
      const __amdgpu_buffer_rsrc_t r_l = mk_rsrc(a.ws_dst[late_h ? lbl2 : x8], WS_USED);
      bool known_d;
      const unsigned t_x = stream_time(ldst, known_d) + (((xlink && !late_h) ? X_XO : late_w ? I_XO - 128u : I_XO) + (unsigned)(wave * 16 + (pi >> 2))) * SLOT256;
      const unsigned xa_d = ln * 16u + (late_w ? WB_OFF + par * WB_PAR + (unsigned)rank * WB_SRC + (unsigned)(wave * 64 + 48 + (pi >> 2)) * 1024u
                                               : XA_OFF + par * XA_PAR + (unsigned)rank * XA_SRC + (unsigned)(wave * 32 + 16 + (pi >> 2)) * 1024u);
      const unsigned sc_d = (pi >> 2) * 64u + (late_w ? EW_OFF + (par * 8 + rank) * 4096u + 3072u : EX_OFF + (par * 8 + rank) * 2048u + 1024u);
      u4 f[4];
#pragma unroll
      for (int u = 0; u < 4; ++u) f[u] = *(const u4*)(smem + LDS_XO + (wave * 4 + u) * 1024 + ln * 16);
      const unsigned ew = cw | (unsigned)(exp_of(sctl[ln]) + 512);
      if (known_o && known_d && (int)(t_o - t_x) < 0) {
        wait_until(t_o);
        st128(r_ow, (row * GN + col) * 4u, o);
        o_sent = true;
      }
#pragma unroll
      for (int u = 0; u < 4; ++u) {
        if (known_d) wait_until(t_x + (unsigned)(4 * u) * SLOT256);
        st128(r_l, xa_d + u * 4096u, f[u]);
      }
      if (late_w) ACK(2); else ACK(3);
      if (wave == 1) st32(r_l, e15 * 4 + ekg * 256, sc_d, ew);
    }
    if (!o_sent) {
      if (known_o) wait_until(t_o);
      STAMP(3);
      st128(r_ow, (row * GN + col) * 4u, o);
      if (ph < 3) ACK(4); else ACK(5);
    }
  } else {
    const unsigned oo = (unsigned)(((32 * (rb - 8) + m) * GN + (rank >> 2) * 1024 + panel * 64 + q * 4) * 4);
    st128(r_out, oo, o);
    STAMP(5);
    if (xlink) {
      const unsigned mark = mark_of(ln);
      for (;;) {
        const u4 t = ld128(r_out, mark, 0), u = ld128(r_out, mark + 2048u, 0);
        if (__all(t.x != EMPTY && t.y != EMPTY && t.z != EMPTY && t.w != EMPTY && u.x != EMPTY && u.y != EMPTY && u.z != EMPTY && u.w != EMPTY)) break;
        if (wt.expired_fast()) {
          WAIT_VM0();
          st32(r_out, oo, 0, QNAN);
          break;
        }
      }
    }
    STAMP(6);
  }
}

extern "C" void kernel_launch(int rank, int nranks, ncclComm_t comm, hipStream_t stream, void* const* d_in,
                              const int* in_sizes, int n_in, void* d_out, int out_size, const int* in_dtypes,
                              int out_dtype, void* const* d_out_all, void* const* d_ws_all, size_t ws_bytes) {
  if (nranks != NR || (unsigned)rank >= (unsigned)NR || ws_bytes < WS_USED || n_in < 2 || in_sizes[0] != 2048 * KS ||
      in_sizes[1] != KS * GN || out_size != 256 * GN) {
    fprintf(stderr, "unexpected problem shape\n");
    return;
  }
  Args a;
  a.x = (const float*)d_in[0];
  a.w = (const float*)d_in[1];
  for (int p = 0; p < NR; ++p) {
    const int d = dst_of(rank, p) & 7;
    a.xf[p] = a.x + (size_t)(1024 * ((d >> 2) ^ 1) + 64 * (d & 3)) * KS;
  }
  for (int p = 0; p < NR; ++p) { a.ws_dst[p] = (char*)d_ws_all[dst_of(rank, p) & 7]; a.out[p] = (float*)d_out_all[p]; }
  a.out[rank] = (float*)d_out;
  a.me = (char*)d_ws_all[rank];
  a.out_me = (float*)d_out;
  a.rank = rank;
  fused_gemm_rs_silu<<<NWG, NTHREADS, 0, stream>>>(a);
}
```

```cpp
#include <hip/hip_runtime.h>
#include <rccl/rccl.h>
#include <cstdio>

typedef _Float16 h8 __attribute__((ext_vector_type(8)));
typedef float f4 __attribute__((ext_vector_type(4)));
typedef unsigned u4 __attribute__((ext_vector_type(4)));
typedef int i4 __attribute__((ext_vector_type(4)));
typedef __attribute__((address_space(1))) unsigned gu32;
typedef __attribute__((address_space(1))) unsigned long long gu64;
typedef __attribute__((address_space(1))) char gchar;

constexpr int NR = 8;
constexpr int GN = 2048;
constexpr int KS = 256;
constexpr int NWG = 256;
constexpr int NW = 8;
constexpr int NTHREADS = NW * 64;

constexpr unsigned WB_SRC = 521u << 10, XA_SRC = 265u << 10;
constexpr unsigned WB_PAR = 8 * WB_SRC;
constexpr unsigned XA_PAR = 8 * XA_SRC;
constexpr unsigned WB_OFF = 0;
constexpr unsigned XA_OFF = WB_OFF + 2 * WB_PAR;
constexpr unsigned EW_OFF = XA_OFF + 2 * XA_PAR;
constexpr unsigned EX_OFF = EW_OFF + 65536;
constexpr unsigned CTR_OFF = EX_OFF + 32768;
constexpr unsigned TE_OFF = CTR_OFF + 32768;
constexpr unsigned TMO_OFF = TE_OFF + 1024;
constexpr unsigned WS_USED = TMO_OFF + 128;
constexpr unsigned OUT_BYTES = 256u * GN * 4u;
constexpr unsigned IN_BYTES = 2048u * KS * 4u;
#define DEADLINE 100000000ull
constexpr unsigned long long DEADLINE_TICKS = DEADLINE;
constexpr int AUX_SYS = 17;
constexpr unsigned EMPTY = 0xffffffffu, QNAN = 0x7fc00000u;
constexpr int POLL_SLEEP = 8, FAST_SLEEP = 2;
constexpr unsigned SLOT256 = 440;
constexpr int RQ = 2;
constexpr unsigned BASE_AHEAD = 4;
constexpr int XSW = 7;
constexpr int SPLITK = 4, SPLITA = 6;
constexpr unsigned SHORT_N = 16 - 3 * RQ, SHORT_LEN = 8 * SHORT_N, HOLE = 32 * RQ;
constexpr unsigned X_XO = 512 + SHORT_LEN, X_OUT = X_XO + 128;
constexpr unsigned I_RELAY = 512, I_XO = 640 + HOLE;

struct Args {
  const float* x;
  const float* w;
  char* me;
  float* out_me;
  int rank;
  const float* xf[NR];
  char* ws_dst[NR];
  float* out[NR];
};
__host__ __device__ inline int dst_of(int rank, int label) {
  const bool cross = label <= 2 || label == 7;
  return ((((rank >> 2) ^ (cross ? 1 : 0)) & 1) << 2) | ((cross ? label : label - 6 + rank) & 3);
}

#define WAIT_VM0() asm volatile("s_waitcnt vmcnt(0)" ::: "memory")
#define WAIT_LGKM0() asm volatile("s_waitcnt lgkmcnt(0)" ::: "memory")
#define COMPILER_FENCE() asm volatile("" ::: "memory")

__device__ __forceinline__ __amdgpu_buffer_rsrc_t mk_rsrc(const void* p, unsigned bytes) {
  return __builtin_amdgcn_make_buffer_rsrc((void*)p, (short)0, (int)bytes, 0x00020000);
}
__device__ __forceinline__ void st128(__amdgpu_buffer_rsrc_t r, unsigned voff, u4 v) {
  __builtin_amdgcn_raw_buffer_store_b128(v, r, (int)voff, 0, AUX_SYS);
  __builtin_amdgcn_sched_barrier(0);
}
__device__ __forceinline__ void st32(__amdgpu_buffer_rsrc_t r, unsigned voff, unsigned soff, unsigned v) {
  __builtin_amdgcn_raw_buffer_store_b32(v, r, (int)voff, (int)soff, AUX_SYS);
}
__device__ __forceinline__ u4 ld128(__amdgpu_buffer_rsrc_t r, unsigned voff, unsigned soff) {
  return __builtin_amdgcn_raw_buffer_load_b128(r, (int)voff, (int)soff, AUX_SYS);
}
__device__ __forceinline__ unsigned ld32(__amdgpu_buffer_rsrc_t r, unsigned voff, unsigned soff) {
  return __builtin_amdgcn_raw_buffer_load_b32(r, (int)voff, (int)soff, AUX_SYS);
}
__device__ __forceinline__ float max3abs(float m, float a, float b) {
  float r;
  asm("v_max3_f32 %0, |%1|, |%2|, %3" : "=v"(r) : "v"(a), "v"(b), "v"(m));
  return r;
}
__device__ __forceinline__ int exp_of(unsigned amax_bits) { return (int)(amax_bits >> 23) - (127 + 14); }
__device__ __forceinline__ u4 pack8(const float (&v)[8], int ne, unsigned tag) {
  h8 h;
  h[0] = (_Float16)0.f;
#pragma unroll
  for (int j = 1; j < 8; ++j) h[j] = (_Float16)__builtin_amdgcn_ldexpf(v[j], ne);
  u4 r = __builtin_bit_cast(u4, h);
  const unsigned t = __builtin_bit_cast(unsigned, __builtin_amdgcn_cvt_pkrtz(__builtin_amdgcn_ldexpf(v[0], ne), 0.f));
  unsigned mag = t & 0x7fffu;
  mag = mag < 0x7ffeu ? mag : 0x7ffeu;
  mag += (mag ^ tag) & 1u;
  r.x |= (t & 0x8000u) | mag;
  return r;
}
__device__ __forceinline__ unsigned now256() { return (unsigned)__builtin_amdgcn_s_memrealtime() << 8; }
__device__ __forceinline__ void wait_until(unsigned t256) {
  int d = (int)(t256 - now256());
  if (d > (1 << 22)) return;
  while (d > 4096) { __builtin_amdgcn_s_sleep(2); d = (int)(t256 - now256()); }
  while (d > 0) d = (int)(t256 - now256());
}

struct Waiter {
  gu32* tmo; unsigned call; unsigned long long t0; bool gave_up;
  __device__ __forceinline__ void start() { t0 = __builtin_amdgcn_s_memrealtime(); gave_up = false; }
  __device__ __forceinline__ bool expired() { return expired_t<POLL_SLEEP>(); }
  __device__ __forceinline__ bool expired_fast() { return expired_t<FAST_SLEEP>(); }
  template <int SLEEP> __device__ __forceinline__ bool expired_t() {
    const unsigned long long dt = __builtin_amdgcn_s_memrealtime() - t0;
    if (dt > 4000) {
      if (dt > DEADLINE_TICKS || __hip_atomic_load(tmo, __ATOMIC_RELAXED, __HIP_MEMORY_SCOPE_AGENT) == call) {
        __hip_atomic_store(tmo, call, __ATOMIC_RELAXED, __HIP_MEMORY_SCOPE_AGENT);
        gave_up = true;
        return true;
      }
    }
    __builtin_amdgcn_s_sleep(SLEEP);
    asm volatile("" ::: "memory");
    return false;
  }
};

constexpr int LDS_PART = 0;
constexpr int LDS_SC = NW * 8192;
constexpr int LDS_XO = LDS_SC + 4096;
constexpr int LDS_CTL = LDS_XO + 32768;
constexpr int LDS_BYTES = LDS_CTL + 512;
constexpr int C_ITEM = 96, C_DEAD = 105, C_GRID = 106;

__device__ __forceinline__ void lds_arrive(unsigned* c, int lane) {
  WAIT_LGKM0();
  if (lane == 0) __hip_atomic_fetch_add(c, 1u, __ATOMIC_RELAXED, __HIP_MEMORY_SCOPE_WORKGROUP);
}
__device__ __forceinline__ void lds_wait(unsigned* c, unsigned n) {
  while (__hip_atomic_load(c, __ATOMIC_RELAXED, __HIP_MEMORY_SCOPE_WORKGROUP) < n) __builtin_amdgcn_s_sleep(1);
  COMPILER_FENCE();
}

__global__ void __launch_bounds__(NTHREADS) fused_gemm_rs_silu(Args a) {
  __shared__ __attribute__((aligned(16))) char smem[LDS_BYTES];
  unsigned* sctl = (unsigned*)(smem + LDS_CTL);
  const int b = blockIdx.x;
  const int lane = threadIdx.x & 63;
  const int wave = __builtin_amdgcn_readfirstlane(threadIdx.x >> 6);
  const int x8 = b & 7, ii = b >> 3, pi = ii & 15;
  const bool foreign = ii >= 16;
  typedef __attribute__((address_space(1))) const f4 gf4;
  gchar* me = (gchar*)a.me;
  gu32* ctr = (gu32*)(me + CTR_OFF) + b * 32;
  f4 xlo = {}, xhi = {};
  if (__builtin_expect(!foreign, 1)) {
    const unsigned xq = (unsigned)((256 * (pi >> 2) + 16 * (pi & 3) + (lane >> 2)) * KS + 32 * (int)(threadIdx.x >> 6) + 4 * (lane & 3)) * 4u;
    gchar* px = (gchar*)a.xf[x8];
    xlo = *(gf4*)(px + xq);
    xhi = *(gf4*)(px + xq + 64);
  }
  const unsigned call_v = __hip_atomic_load(ctr, __ATOMIC_RELAXED, __HIP_MEMORY_SCOPE_AGENT);
  __builtin_amdgcn_sched_barrier(0);
  const float* p_x = a.x; const float* p_w = a.w; float* p_out = a.out_me; char* p_dst = a.ws_dst[x8];
  int rank = a.rank;
  asm volatile("" : "+s"(p_x), "+s"(p_w), "+s"(p_out), "+s"(p_dst), "+s"(rank));
  const int l15 = lane & 15, kg = lane >> 4;
  const unsigned l16 = lane * 16;
  const int ph = pi & 3;
  const int panel = ph * 4 + (x8 >> 1);
  const int rb = (foreign ? 0 : 8) + 4 * (x8 & 1) + (pi >> 2);
  const int dst = dst_of(rank, x8);
  const bool xlink = ((dst ^ rank) & 4) != 0;
  const int src = wave;
  const __amdgpu_buffer_rsrc_t r_me = mk_rsrc((char*)me, WS_USED);
  const __amdgpu_buffer_rsrc_t r_out = mk_rsrc(p_out, OUT_BYTES);
  const __amdgpu_buffer_rsrc_t r_d = mk_rsrc(p_dst, WS_USED);
  const __amdgpu_buffer_rsrc_t r_x = mk_rsrc(p_x, IN_BYTES);
  const bool late_w = foreign && ph == 0 && !xlink;
  const bool late_h = foreign && ph == 1 && xlink;
  const bool late = foreign && ph < 2;
  const int lbl2 = 3 + (x8 & 3);
  const int ldst = late_h ? dst_of(rank, lbl2) : dst;
  auto mark_of = [&](int ln) {
    return (unsigned)(((64 * (dst & 3) + 16 * (pi & 3) + 2 * wave + ((ln >> 4) & 1)) * GN + (dst >> 2) * 1024 +
                       ((ln >> 5) * 4 + (pi >> 2)) * 64 + (ln & 15) * 4) * 4);
  };
  unsigned call, par, tag, cw;
  auto set_call = [&](unsigned call_v) {
    call = __builtin_amdgcn_readfirstlane(call_v) + 1u;
    par = call & 1u; tag = ((call + 1u) >> 1) & 1u;
    cw = call << 10;
  };
  auto stream_time = [&](unsigned d, bool& known) {
    const unsigned long long te = __hip_atomic_load((gu64*)(me + TE_OFF + d * 128u), __ATOMIC_RELAXED, __HIP_MEMORY_SCOPE_AGENT);
    known = __builtin_amdgcn_readfirstlane((unsigned)(te >> 32)) == call;
    return (unsigned)__builtin_amdgcn_readfirstlane((unsigned)te);
  };
  auto max_only = [&](int it, const float (&d)[8]) {
    float m = max3abs(0.f, d[0], d[1]);
    m = max3abs(m, d[2], d[3]); m = max3abs(m, d[4], d[5]); m = max3abs(m, d[6], d[7]);
    __hip_atomic_fetch_max(&sctl[16 * it + l15], __float_as_uint(m), __ATOMIC_RELAXED, __HIP_MEMORY_SCOPE_WORKGROUP);
  };

  if (__builtin_expect(!foreign, 1)) {
    if (wave == 0) { sctl[lane] = 0; sctl[64 + lane] = 0; WAIT_LGKM0(); }
    __builtin_amdgcn_s_barrier();
    float v[2][8];
    const __amdgpu_buffer_rsrc_t r_w = mk_rsrc(p_w, IN_BYTES);
    const unsigned wo = (unsigned)((32 * wave + 4 * kg) * GN + (dst >> 2) * 1024 + pi * 16 + l15) * 4u;
    const bool xr = xlink;
    auto phase_of = [&](int it) { return (unsigned)(xr ? (it == 1 ? 0 : it == 2 ? 3 : it - 2) : it - 1); };
    auto load_w = [&](int it, float (&d)[8]) {
      const unsigned o = wo + phase_of(it) * 1024u;
#pragma unroll
      for (int j = 0; j < 8; ++j)
        d[j] = __builtin_bit_cast(float, __builtin_amdgcn_raw_buffer_load_b32(r_w, o, (16 * (j >> 2) + (j & 3)) * GN * 4, 0));
    };
    load_w(1, v[1]);
    unsigned grid;
    {
      const unsigned now = now256();
      grid = now - ((now & 0x3fffffffu) + 16u * SLOT256 - (unsigned)pi * SLOT256) % (16u * SLOT256);
      asm volatile("" : "+s"(grid));
    }
    __builtin_amdgcn_sched_barrier(0);
    {
      float m = max3abs(0.f, xlo.x, xlo.y);
      m = max3abs(m, xlo.z, xlo.w); m = max3abs(m, xhi.x, xhi.y); m = max3abs(m, xhi.z, xhi.w);
      __hip_atomic_fetch_max(&sctl[lane >> 2], __float_as_uint(m), __ATOMIC_RELAXED, __HIP_MEMORY_SCOPE_WORKGROUP);
      char* t = smem + LDS_PART + wave * 8192 + ((lane >> 2) + 17 * (lane & 3)) * 16;
      *(f4*)t = xlo; *(f4*)(t + 2048) = xhi;
    }
    WAIT_LGKM0();
    __builtin_amdgcn_s_barrier();
    COMPILER_FENCE();
    {
      const char* t = smem + LDS_PART + wave * 8192 + (l15 + 17 * kg) * 16;
      const f4 lo = *(const f4*)t, hi = *(const f4*)(t + 2048);
      v[0][0] = lo.x; v[0][1] = lo.y; v[0][2] = lo.z; v[0][3] = lo.w; v[0][4] = hi.x; v[0][5] = hi.y; v[0][6] = hi.z; v[0][7] = hi.w;
    }
    set_call(call_v);
    auto max_item = [&](int it, const float (&d)[8]) { max_only(it, d); lds_arrive(&sctl[C_ITEM + it], lane); };
    const unsigned xa_d = XA_OFF + par * XA_PAR + (unsigned)rank * XA_SRC + (unsigned)(wave * 32 + pi) * 1024u + l16;
    const unsigned wb_d = WB_OFF + par * WB_PAR + (unsigned)rank * WB_SRC + (unsigned)(wave * 64 + pi) * 1024u + l16;
    const unsigned ex_d = EX_OFF + (par * 8 + rank) * 2048u + pi * 64u, ew_d = EW_OFF + (par * 8 + rank) * 4096u + pi * 64u;
    const int ri = pi - (16 - 4 * RQ);
    const bool send3 = !xr || ri < 0 || ri / RQ == (dst & 3);
    const int idx3 = ri < 0 ? pi : 16 - 4 * RQ + ri % RQ;
#pragma unroll
    for (int it = 0; it < 5; ++it) {
      if (it == 4 && !xlink) break;
      if (it) lds_wait(&sctl[C_ITEM + it], NW);
      const int e = exp_of(__hip_atomic_load(&sctl[16 * it + l15], __ATOMIC_RELAXED, __HIP_MEMORY_SCOPE_WORKGROUP));
      u4 f = pack8(v[it & 1], -e, tag);
      asm volatile("" : "+v"(f));
      if (it == 0) {
        if (wave == 0) {
          unsigned long long t;
          asm volatile("s_memrealtime %0\n\ts_waitcnt lgkmcnt(0)" : "=s"(t) : "v"(f));
          const unsigned now = (unsigned)t << 8;
          while ((int)(now - grid) > 0) grid += 16u * SLOT256;
          grid |= 1u;
          if (lane == 0) {
            __hip_atomic_store(&sctl[C_GRID], grid, __ATOMIC_RELAXED, __HIP_MEMORY_SCOPE_WORKGROUP);
            if (pi == 0)
              __hip_atomic_store((gu64*)(me + TE_OFF + dst * 128u), ((unsigned long long)call << 32) | (now + BASE_AHEAD * SLOT256),
                                 __ATOMIC_RELAXED, __HIP_MEMORY_SCOPE_AGENT);
          }
        } else {
          lds_wait(&sctl[C_GRID], 1u);
          grid = __builtin_amdgcn_readfirstlane(__hip_atomic_load(&sctl[C_GRID], __ATOMIC_RELAXED, __HIP_MEMORY_SCOPE_WORKGROUP));
        }
        if (wave >= 4) max_item(1, v[1]);
      }
      int slot = it * 128 + wave * 16;
      if (xr) { if (it == 2) slot = 256 + wave * (int)SHORT_N + idx3 - pi; else if (it > 2) slot += (int)SHORT_LEN - 128; }
      wait_until(grid + (unsigned)slot * SLOT256);
      if (it != 2 || send3) { st128(r_d, it == 0 ? xa_d : wb_d + phase_of(it) * 16384u, f); }
      if (wave == 1 && lane < 16) st32(r_d, lane * 4, it == 0 ? ex_d : ew_d + phase_of(it) * 1024u, cw | (unsigned)(e + 512));
      if (it == 0) {
        if (xlink) {
          const unsigned mark = mark_of(lane);
          st128(r_out, mark, u4{EMPTY, EMPTY, EMPTY, EMPTY});
          st128(r_out, mark + 2048u, u4{EMPTY, EMPTY, EMPTY, EMPTY});
        }
      }
      if (it == 1 && wave == 0 && lane == 0) __hip_atomic_store(ctr, call, __ATOMIC_RELAXED, __HIP_MEMORY_SCOPE_AGENT);
      if (it + 2 < 4 || (it == 2 && xlink)) {
        load_w(it + 2, v[it & 1]);
        if (it == 0) { if (wave < 4) max_item(1, v[1]); WAIT_VM0(); }
        max_item(it + 2, v[it & 1]);
        if (it == 0) {
          bool known;
          const unsigned g0 = stream_time(dst, known);
          if (known) grid = g0 + (unsigned)pi * SLOT256;
        }
      }
    }
    wait_until(grid + ((xlink ? X_XO : I_RELAY) - 16u) * SLOT256 + 2048u);
  } else {
    if (wave == 0) { sctl[lane] = 0; sctl[64 + lane] = 0; WAIT_LGKM0(); }
    set_call(call_v);
    WAIT_VM0();
    __builtin_amdgcn_s_barrier();
    if (wave == 0 && lane == 0) __hip_atomic_store(ctr, call, __ATOMIC_RELAXED, __HIP_MEMORY_SCOPE_AGENT);
    if (late) {
      __builtin_amdgcn_s_sleep(32);
      const unsigned xo = (unsigned)(l15 * KS + 32 * wave + 4 * kg) * 4u;
      float v[4][8];
      if (late_w) {
        const __amdgpu_buffer_rsrc_t r_w = mk_rsrc(p_w, IN_BYTES);
        const unsigned wo = (unsigned)((32 * wave + 4 * kg) * GN + (dst >> 2) * 1024 + (48 + (pi >> 2)) * 16 + l15) * 4u;
#pragma unroll
        for (int u = 0; u < 4; ++u)
#pragma unroll
          for (int j = 0; j < 8; ++j)
            v[u][j] = __builtin_bit_cast(float, __builtin_amdgcn_raw_buffer_load_b32(r_w, wo + u * 256, (16 * (j >> 2) + (j & 3)) * GN * 4, 0));
      } else {
#pragma unroll
        for (int u = 0; u < 4; ++u) {
          const int row = 256 * ldst + 16 * ((pi >> 2) + 4 * u);
          const f4 lo = __builtin_bit_cast(f4, __builtin_amdgcn_raw_buffer_load_b128(r_x, xo, row * KS * 4, 0));
          const f4 hi = __builtin_bit_cast(f4, __builtin_amdgcn_raw_buffer_load_b128(r_x, xo + 64, row * KS * 4, 0));
          v[u][0] = lo.x; v[u][1] = lo.y; v[u][2] = lo.z; v[u][3] = lo.w; v[u][4] = hi.x; v[u][5] = hi.y; v[u][6] = hi.z; v[u][7] = hi.w;
        }
      }
#pragma unroll
      for (int u = 0; u < 4; ++u) max_only(u, v[u]);
      WAIT_LGKM0();
      __builtin_amdgcn_s_barrier();
      COMPILER_FENCE();
#pragma unroll
      for (int u = 0; u < 4; ++u) {
        const int e = exp_of(__hip_atomic_load(&sctl[16 * u + l15], __ATOMIC_RELAXED, __HIP_MEMORY_SCOPE_WORKGROUP));
        *(u4*)(smem + LDS_XO + (wave * 4 + u) * 1024 + l16) = pack8(v[u], -e, tag);
      }
    }
  }

  Waiter wt; wt.tmo = (gu32*)(me + TMO_OFF); wt.call = call; wt.start();
  const unsigned wb_base = WB_OFF + par * WB_PAR + (unsigned)src * WB_SRC + (unsigned)(panel * 4) * 1024u;
  const unsigned xa_base = XA_OFF + par * XA_PAR + (unsigned)src * XA_SRC + (unsigned)(rb * 2) * 1024u;
  const unsigned tagm = tag ? 0u : ~0u;
  int* sc = (int*)(smem + LDS_SC + wave * 512);
  const unsigned hoff = (unsigned)((src + (lane & 3) * 2) & 7) * 128u + 112u;

  u4 A[8][2], B[8][4];
  f4 acc[2][4];
#pragma unroll
  for (int mi = 0; mi < 2; ++mi)
#pragma unroll
    for (int ni = 0; ni < 4; ++ni) acc[mi][ni] = f4{0.f, 0.f, 0.f, 0.f};
#define PROBE_A(kb) do { for (;;) { const unsigned pv = ld32(r_me, (lane & 1) * 1024u + hoff, xa_base + (kb) * 32768u); \
      if (__all((pv ^ tagm) & 1u) || wt.expired()) break; } COMPILER_FENCE(); } while (0)
#define PROBE_B(kb) do { for (;;) { const unsigned pv = ld32(r_me, (lane & 3) * 1024u + hoff, wb_base + (kb) * 65536u); \
      if (__all((pv ^ tagm) & 1u) || wt.expired()) break; } COMPILER_FENCE(); } while (0)
#define ISSUE_A(k0, k1) _Pragma("unroll") for (int kb = (k0); kb < (k1); ++kb) _Pragma("unroll") for (int mi = 0; mi < 2; ++mi) \
      A[kb][mi] = ld128(r_me, l16 + mi * 1024u, xa_base + kb * 32768u)
#define ISSUE_B(k0, k1) _Pragma("unroll") for (int kb = (k0); kb < (k1); ++kb) _Pragma("unroll") for (int ni = 0; ni < 4; ++ni) \
      B[kb][ni] = ld128(r_me, l16 + ni * 1024u, wb_base + kb * 65536u)
#define CHECK_A(k0, k1) _Pragma("unroll") for (int kb = (k0); kb < (k1); ++kb) _Pragma("unroll") for (int mi = 0; mi < 2; ++mi) ok &= A[kb][mi].x ^ tagm
#define CHECK_B(k0, k1) _Pragma("unroll") for (int kb = (k0); kb < (k1); ++kb) _Pragma("unroll") for (int ni = 0; ni < 4; ++ni) ok &= B[kb][ni].x ^ tagm
#define MMA(k0, k1) _Pragma("unroll") for (int kb = (k0); kb < (k1); ++kb) _Pragma("unroll") for (int mi = 0; mi < 2; ++mi) \
      _Pragma("unroll") for (int ni = 0; ni < 4; ++ni) \
        acc[mi][ni] = __builtin_amdgcn_mfma_f32_16x16x32_f16(__builtin_bit_cast(h8, B[kb][ni]), __builtin_bit_cast(h8, A[kb][mi]), acc[mi][ni], 0, 0, 0)
  auto ld_ex = [&]() { return ld32(r_me, (lane & 31) * 4, EX_OFF + (par * 8 + src) * 2048u + rb * 128u); };
  auto ld_ew = [&]() { return ld32(r_me, lane * 4, EW_OFF + (par * 8 + src) * 4096u + panel * 256u); };

  if (foreign) {
    PROBE_A(7);
    for (;;) {
      unsigned ok = 1u;
      ISSUE_A(0, 8);
      const unsigned exv = ld_ex();
      CHECK_A(0, 8);
      sc[64 + lane] = (int)(exv & 1023u) - 1024;
      if (__all((ok & 1u) && (exv ^ cw) < 1024u) || wt.expired()) break;
    }
    if (ph == 3 && !xlink && dst != rank) {
      const unsigned fo = WB_OFF + par * WB_PAR + (unsigned)((((rank >> 2) ^ 1) << 2) | (pi >> 2)) * WB_SRC +
                          (unsigned)(wave * 64 + 64 - 4 * RQ + (rank & 3) * RQ) * 1024u + l16;
      bool known;
      const unsigned te = stream_time(dst, known);
      if (known) wait_until(te + (256u + SHORT_LEN) * SLOT256);
      u4 f[RQ];
      for (;;) {
        unsigned ok = 1u;
#pragma unroll
        for (int u = 0; u < RQ; ++u) { f[u] = ld128(r_me, fo + u * 1024u, 0); }
#pragma unroll
        for (int u = 0; u < RQ; ++u) ok &= f[u].x ^ tagm;
        if (__all(ok & 1u) || wt.expired()) break;
      }
#pragma unroll
      for (int u = 0; u < RQ; ++u) {
        if (known) wait_until(te + (I_RELAY + (unsigned)(wave * 4 * RQ + u * 4 + (pi >> 2))) * SLOT256);
        st128(r_d, fo + u * 1024u, f[u]);
      }
    }
    PROBE_B(SPLITK - 1);
    for (;;) {
      unsigned ok = 1u;
      ISSUE_B(0, SPLITK);
      CHECK_B(0, SPLITK);
      if (__all(ok & 1u) || wt.expired()) break;
    }
    MMA(0, SPLITK);
    PROBE_B(7);
    for (;;) {
      unsigned ok = 1u;
      ISSUE_B(SPLITK, 8);
      const unsigned ewv = ld_ew();
      CHECK_B(SPLITK, 8);
      sc[lane] = (int)(ewv & 1023u);
      if (__all((ok & 1u) && (ewv ^ cw) < 1024u) || wt.expired()) break;
    }
    MMA(SPLITK, 8);
  } else {
    PROBE_B(7);
    bool a1, a2;
    {
      unsigned ok = 1u;
      ISSUE_B(0, 8);
      const unsigned ewv = ld_ew();
      ISSUE_A(0, 8);
      const unsigned exv = ld_ex();
      CHECK_B(0, 8);
      sc[lane] = (int)(ewv & 1023u);
      bool bok = __all((ok & 1u) && (ewv ^ cw) < 1024u);
      ok = 1u;
      CHECK_A(0, SPLITA);
      a1 = __all(ok & 1u);
      ok = 1u;
      CHECK_A(SPLITA, 8);
      sc[64 + lane] = (int)(exv & 1023u) - 1024;
      a2 = __all((ok & 1u) && (exv ^ cw) < 1024u);
      while (!bok && !wt.expired()) {
        ok = 1u;
        ISSUE_B(0, 8);
        const unsigned ew2 = ld_ew();
        CHECK_B(0, 8);
        sc[lane] = (int)(ew2 & 1023u);
        bok = __all((ok & 1u) && (ew2 ^ cw) < 1024u);
      }
    }
    if (!a1) {
      PROBE_A(SPLITA - 1);
      for (;;) {
        unsigned ok = 1u;
        ISSUE_A(0, SPLITA);
        CHECK_A(0, SPLITA);
        if (__all(ok & 1u) || wt.expired()) break;
      }
    }
    MMA(0, SPLITA);
    if (!a2) {
      for (;;) {
        unsigned ok = 1u;
        ISSUE_A(SPLITA, 8);
        const unsigned exv = ld_ex();
        CHECK_A(SPLITA, 8);
        sc[64 + lane] = (int)(exv & 1023u) - 1024;
        if (__all((ok & 1u) && (exv ^ cw) < 1024u) || wt.expired_fast()) break;
      }
    }
    MMA(SPLITA, 8);
  }

  int ln = lane;
  asm volatile("" : "+v"(ln));
  const int e15 = ln & 15, ekg = ln >> 4;
  {
    WAIT_LGKM0();
    int exv[2];
    i4 ewv[4];
#pragma unroll
    for (int mi = 0; mi < 2; ++mi) exv[mi] = sc[64 + mi * 16 + e15];
#pragma unroll
    for (int ni = 0; ni < 4; ++ni) ewv[ni] = *(const i4*)(sc + ni * 16 + ekg * 4);
#pragma unroll
    for (int mi = 0; mi < 2; ++mi)
#pragma unroll
      for (int ni = 0; ni < 4; ++ni) {
        f4 o;
        o.x = __builtin_amdgcn_ldexpf(acc[mi][ni].x, exv[mi] + ewv[ni].x);
        o.y = __builtin_amdgcn_ldexpf(acc[mi][ni].y, exv[mi] + ewv[ni].y);
        o.z = __builtin_amdgcn_ldexpf(acc[mi][ni].z, exv[mi] + ewv[ni].z);
        o.w = __builtin_amdgcn_ldexpf(acc[mi][ni].w, exv[mi] + ewv[ni].w);
        const int q = ni * 4 + ekg;
        *(f4*)(smem + LDS_PART + wave * 8192 + ((mi * 4 + ni) * 64 + (ln & 48) + (e15 ^ q)) * 16) = o;
      }
  }
  if (wt.gave_up && lane == 0) __hip_atomic_store(&sctl[C_DEAD], 1u, __ATOMIC_RELAXED, __HIP_MEMORY_SCOPE_WORKGROUP);
  WAIT_LGKM0();
  __builtin_amdgcn_s_barrier();
  COMPILER_FENCE();
  const bool dead = __hip_atomic_load(&sctl[C_DEAD], __ATOMIC_RELAXED, __HIP_MEMORY_SCOPE_WORKGROUP) != 0;
  const int m = wave * 4 + ekg, q = e15;
  const int idx = ((m >> 4) * 4 + (q >> 2)) * 64 + (q & 3) * 16 + ((m & 15) ^ q);
  f4 s = *(const f4*)(smem + LDS_PART + idx * 16);
#pragma unroll
  for (int w = 1; w < NW; ++w) s += *(const f4*)(smem + LDS_PART + w * 8192 + idx * 16);
  u4 o;
#pragma unroll
  for (int e = 0; e < 4; ++e) {
    const float y = s[e];
    const float c = __builtin_fminf(__builtin_fmaxf(y, -60.f), 60.f);
    const unsigned r = __float_as_uint(y * __builtin_amdgcn_rcpf(1.f + __expf(-c)));
    o[e] = (dead || r == EMPTY) ? QNAN : r;
  }
  if (foreign) {
    const unsigned owner = (unsigned)((((rank >> 2) ^ 1) << 2) | (rb >> 1));
    const unsigned row = (unsigned)(64 * (rank & 3) + 32 * (rb & 1) + m);
    const unsigned col = (unsigned)((rank >> 2) * 1024 + panel * 64 + q * 4);
    const __amdgpu_buffer_rsrc_t r_ow = mk_rsrc(a.out[owner & 7], OUT_BYTES);
    asm volatile("" : "+v"(o));
    bool known_o;
    const int oc = xlink ? 0 : 1, oi = x8 >= (xlink ? 2 : 5) ? 1 : 0;
    const int ok_ = (ph < 3 ? oc * 3 + ph : 6 + oc) * 4 + oi * 2 + (rb & 1);
    const unsigned t_o = stream_time(owner, known_o) + (X_OUT + (unsigned)(ok_ * 8 + wave)) * SLOT256;
    bool o_sent = false;
    if (late && !(ph == 1 && (wave >= XSW) != late_h)) {
      const __amdgpu_buffer_rsrc_t r_l = mk_rsrc(a.ws_dst[late_h ? lbl2 : x8], WS_USED);
      bool known_d;
      const unsigned t_x = stream_time(ldst, known_d) + (((xlink && !late_h) ? X_XO : late_w ? I_XO - 128u : I_XO) + (unsigned)(wave * 16 + (pi >> 2))) * SLOT256;
      const unsigned xa_d = ln * 16u + (late_w ? WB_OFF + par * WB_PAR + (unsigned)rank * WB_SRC + (unsigned)(wave * 64 + 48 + (pi >> 2)) * 1024u
                                               : XA_OFF + par * XA_PAR + (unsigned)rank * XA_SRC + (unsigned)(wave * 32 + 16 + (pi >> 2)) * 1024u);
      const unsigned sc_d = (pi >> 2) * 64u + (late_w ? EW_OFF + (par * 8 + rank) * 4096u + 3072u : EX_OFF + (par * 8 + rank) * 2048u + 1024u);
      u4 f[4];
#pragma unroll
      for (int u = 0; u < 4; ++u) f[u] = *(const u4*)(smem + LDS_XO + (wave * 4 + u) * 1024 + ln * 16);
      const unsigned ew = cw | (unsigned)(exp_of(sctl[ln]) + 512);
      if (known_o && known_d && (int)(t_o - t_x) < 0) {
        wait_until(t_o);
        st128(r_ow, (row * GN + col) * 4u, o);
        o_sent = true;
      }
#pragma unroll
      for (int u = 0; u < 4; ++u) {
        if (known_d) wait_until(t_x + (unsigned)(4 * u) * SLOT256);
        st128(r_l, xa_d + u * 4096u, f[u]);
      }
      if (wave == 1) st32(r_l, e15 * 4 + ekg * 256, sc_d, ew);
    }
    if (!o_sent) {
      if (known_o) wait_until(t_o);
      st128(r_ow, (row * GN + col) * 4u, o);
    }
  } else {
    const unsigned oo = (unsigned)(((32 * (rb - 8) + m) * GN + (rank >> 2) * 1024 + panel * 64 + q * 4) * 4);
    st128(r_out, oo, o);
    if (xlink) {
      const unsigned mark = mark_of(ln);
      for (;;) {
        const u4 t = ld128(r_out, mark, 0), u = ld128(r_out, mark + 2048u, 0);
        if (__all(t.x != EMPTY && t.y != EMPTY && t.z != EMPTY && t.w != EMPTY && u.x != EMPTY && u.y != EMPTY && u.z != EMPTY && u.w != EMPTY)) break;
        if (wt.expired_fast()) {
          WAIT_VM0();
          st32(r_out, oo, 0, QNAN);
          break;
        }
      }
    }
  }
}

extern "C" void kernel_launch(int rank, int nranks, ncclComm_t comm, hipStream_t stream, void* const* d_in,
                              const int* in_sizes, int n_in, void* d_out, int out_size, const int* in_dtypes,
                              int out_dtype, void* const* d_out_all, void* const* d_ws_all, size_t ws_bytes) {
  if (nranks != NR || (unsigned)rank >= (unsigned)NR || ws_bytes < WS_USED || n_in < 2 || in_sizes[0] != 2048 * KS ||
      in_sizes[1] != KS * GN || out_size != 256 * GN) {
    fprintf(stderr, "unexpected problem shape\n");
    return;
  }
  Args a;
  a.x = (const float*)d_in[0];
  a.w = (const float*)d_in[1];
  for (int p = 0; p < NR; ++p) {
    const int d = dst_of(rank, p) & 7;
    a.xf[p] = a.x + (size_t)(1024 * ((d >> 2) ^ 1) + 64 * (d & 3)) * KS;
  }
  for (int p = 0; p < NR; ++p) { a.ws_dst[p] = (char*)d_ws_all[dst_of(rank, p) & 7]; a.out[p] = (float*)d_out_all[p]; }
  a.out[rank] = (float*)d_out;
  a.me = (char*)d_ws_all[rank];
  a.out_me = (float*)d_out;
  a.rank = rank;
  fused_gemm_rs_silu<<<NWG, NTHREADS, 0, stream>>>(a);
}
```

```cpp
#include <hip/hip_runtime.h>
#include <rccl/rccl.h>
#include <cstdio>

typedef _Float16 h8 __attribute__((ext_vector_type(8)));
typedef float f4 __attribute__((ext_vector_type(4)));
typedef unsigned u4 __attribute__((ext_vector_type(4)));
typedef int i4 __attribute__((ext_vector_type(4)));
typedef __attribute__((address_space(1))) unsigned gu32;
typedef __attribute__((address_space(1))) unsigned long long gu64;
typedef __attribute__((address_space(1))) char gchar;

constexpr int NR = 8;
constexpr int GN = 2048;
constexpr int KS = 256;
constexpr int NWG = 256;
constexpr int NW = 8;
constexpr int NTHREADS = NW * 64;

constexpr unsigned WB_SRC = 521u << 10, XA_SRC = 265u << 10;
constexpr unsigned WB_PAR = 8 * WB_SRC;
constexpr unsigned XA_PAR = 8 * XA_SRC;
constexpr unsigned WB_OFF = 0;
constexpr unsigned XA_OFF = WB_OFF + 2 * WB_PAR;
constexpr unsigned EW_OFF = XA_OFF + 2 * XA_PAR;
constexpr unsigned EX_OFF = EW_OFF + 65536;
constexpr unsigned CTR_OFF = EX_OFF + 32768;
constexpr unsigned TE_OFF = CTR_OFF + 32768;
constexpr unsigned TMO_OFF = TE_OFF + 1024;
constexpr unsigned WS_USED = TMO_OFF + 128;
constexpr unsigned OUT_BYTES = 256u * GN * 4u;
constexpr unsigned IN_BYTES = 2048u * KS * 4u;
#define DEADLINE 100000000ull
constexpr unsigned long long DEADLINE_TICKS = DEADLINE;
constexpr int AUX_SYS = 17;
constexpr unsigned EMPTY = 0xffffffffu, QNAN = 0x7fc00000u;
constexpr int POLL_SLEEP = 8, FAST_SLEEP = 2;
constexpr unsigned SLOT256 = 440;
constexpr int RQ = 2;
constexpr unsigned BASE_AHEAD = 4;
constexpr int XSW = 7;
constexpr int SPLITK = 4, SPLITA = 6;
constexpr unsigned SHORT_N = 16 - 3 * RQ, SHORT_LEN = 8 * SHORT_N, HOLE = 32 * RQ;
constexpr unsigned X_XO = 512 + SHORT_LEN, X_OUT = X_XO + 128;
constexpr unsigned I_RELAY = 512, I_XO = 640 + HOLE;

struct Args {
  const float* x;
  const float* w;
  char* me;
  float* out_me;
  int rank;
  const float* xf[NR];
  char* ws_dst[NR];
  float* out[NR];
};
__host__ __device__ inline int dst_of(int rank, int label) {
  const bool cross = label <= 2 || label == 7;
  return ((((rank >> 2) ^ (cross ? 1 : 0)) & 1) << 2) | ((cross ? label : label - 6 + rank) & 3);
}

#define WAIT_VM0() asm volatile("s_waitcnt vmcnt(0)" ::: "memory")
#define WAIT_LGKM0() asm volatile("s_waitcnt lgkmcnt(0)" ::: "memory")
#define COMPILER_FENCE() asm volatile("" ::: "memory")

__device__ __forceinline__ __amdgpu_buffer_rsrc_t mk_rsrc(const void* p, unsigned bytes) {
  return __builtin_amdgcn_make_buffer_rsrc((void*)p, (short)0, (int)bytes, 0x00020000);
}
__device__ __forceinline__ void st128(__amdgpu_buffer_rsrc_t r, unsigned voff, u4 v) {
  __builtin_amdgcn_raw_buffer_store_b128(v, r, (int)voff, 0, AUX_SYS);
  __builtin_amdgcn_sched_barrier(0);
}
__device__ __forceinline__ void st32(__amdgpu_buffer_rsrc_t r, unsigned voff, unsigned soff, unsigned v) {
  __builtin_amdgcn_raw_buffer_store_b32(v, r, (int)voff, (int)soff, AUX_SYS);
}
__device__ __forceinline__ u4 ld128(__amdgpu_buffer_rsrc_t r, unsigned voff, unsigned soff) {
  return __builtin_amdgcn_raw_buffer_load_b128(r, (int)voff, (int)soff, AUX_SYS);
}
__device__ __forceinline__ unsigned ld32(__amdgpu_buffer_rsrc_t r, unsigned voff, unsigned soff) {
  return __builtin_amdgcn_raw_buffer_load_b32(r, (int)voff, (int)soff, AUX_SYS);
}
__device__ __forceinline__ float max3abs(float m, float a, float b) {
  float r;
  asm("v_max3_f32 %0, |%1|, |%2|, %3" : "=v"(r) : "v"(a), "v"(b), "v"(m));
  return r;
}
__device__ __forceinline__ int exp_of(unsigned amax_bits) { return (int)(amax_bits >> 23) - (127 + 14); }
__device__ __forceinline__ u4 pack8(const float (&v)[8], int ne, unsigned tag) {
  h8 h;
  h[0] = (_Float16)0.f;
#pragma unroll
  for (int j = 1; j < 8; ++j) h[j] = (_Float16)__builtin_amdgcn_ldexpf(v[j], ne);
  u4 r = __builtin_bit_cast(u4, h);
  const unsigned t = __builtin_bit_cast(unsigned, __builtin_amdgcn_cvt_pkrtz(__builtin_amdgcn_ldexpf(v[0], ne), 0.f));
  unsigned mag = t & 0x7fffu;
  mag = mag < 0x7ffeu ? mag : 0x7ffeu;
  mag += (mag ^ tag) & 1u;
  r.x |= (t & 0x8000u) | mag;
  return r;
}
__device__ __forceinline__ unsigned now256() { return (unsigned)__builtin_amdgcn_s_memrealtime() << 8; }
__device__ __forceinline__ void wait_until(unsigned t256) {
  int d = (int)(t256 - now256());
  if (d > (1 << 22)) return;
  while (d > 4096) { __builtin_amdgcn_s_sleep(2); d = (int)(t256 - now256()); }
  while (d > 0) d = (int)(t256 - now256());
}

struct Waiter {
  gu32* tmo; unsigned call; unsigned long long t0; bool gave_up;
  __device__ __forceinline__ void start() { t0 = __builtin_amdgcn_s_memrealtime(); gave_up = false; }
  __device__ __forceinline__ bool expired() { return expired_t<POLL_SLEEP>(); }
  __device__ __forceinline__ bool expired_fast() { return expired_t<FAST_SLEEP>(); }
  template <int SLEEP> __device__ __forceinline__ bool expired_t() {
    const unsigned long long dt = __builtin_amdgcn_s_memrealtime() - t0;
    if (dt > 4000) {
      if (dt > DEADLINE_TICKS || __hip_atomic_load(tmo, __ATOMIC_RELAXED, __HIP_MEMORY_SCOPE_AGENT) == call) {
        __hip_atomic_store(tmo, call, __ATOMIC_RELAXED, __HIP_MEMORY_SCOPE_AGENT);
        gave_up = true;
        return true;
      }
    }
    __builtin_amdgcn_s_sleep(SLEEP);
    asm volatile("" ::: "memory");
    return false;
  }
};

constexpr int LDS_PART = 0;
constexpr int LDS_SC = NW * 8192;
constexpr int LDS_XO = LDS_SC + 4096;
constexpr int LDS_CTL = LDS_XO + 32768;
constexpr int LDS_BYTES = LDS_CTL + 512;
constexpr int C_ITEM = 96, C_DEAD = 105, C_GRID = 106;

__device__ __forceinline__ void lds_arrive(unsigned* c, int lane) {
  WAIT_LGKM0();
  if (lane == 0) __hip_atomic_fetch_add(c, 1u, __ATOMIC_RELAXED, __HIP_MEMORY_SCOPE_WORKGROUP);
}
__device__ __forceinline__ void lds_wait(unsigned* c, unsigned n) {
  while (__hip_atomic_load(c, __ATOMIC_RELAXED, __HIP_MEMORY_SCOPE_WORKGROUP) < n) __builtin_amdgcn_s_sleep(1);
  COMPILER_FENCE();
}

__global__ void __launch_bounds__(NTHREADS) fused_gemm_rs_silu(Args a) {
  __shared__ __attribute__((aligned(16))) char smem[LDS_BYTES];
  unsigned* sctl = (unsigned*)(smem + LDS_CTL);
  const int b = blockIdx.x;
  const int lane = threadIdx.x & 63;
  const int wave = __builtin_amdgcn_readfirstlane(threadIdx.x >> 6);
  const int x8 = b & 7, ii = b >> 3, pi = ii & 15;
  const bool foreign = ii >= 16;
  typedef __attribute__((address_space(1))) const f4 gf4;
  gchar* me = (gchar*)a.me;
  gu32* ctr = (gu32*)(me + CTR_OFF) + b * 32;
  f4 xlo = {}, xhi = {};
  if (__builtin_expect(!foreign, 1)) {
    const unsigned xq = (unsigned)((256 * (pi >> 2) + 16 * (pi & 3) + (lane >> 2)) * KS + 32 * (int)(threadIdx.x >> 6) + 4 * (lane & 3)) * 4u;
    gchar* px = (gchar*)a.xf[x8];
    xlo = *(gf4*)(px + xq);
    xhi = *(gf4*)(px + xq + 64);
  }
  const unsigned call_v = __hip_atomic_load(ctr, __ATOMIC_RELAXED, __HIP_MEMORY_SCOPE_AGENT);
  __builtin_amdgcn_sched_barrier(0);
  const float* p_x = a.x; const float* p_w = a.w; float* p_out = a.out_me; char* p_dst = a.ws_dst[x8];
  int rank = a.rank;
  asm volatile("" : "+s"(p_x), "+s"(p_w), "+s"(p_out), "+s"(p_dst), "+s"(rank));
  const int l15 = lane & 15, kg = lane >> 4;
  const unsigned l16 = lane * 16;
  const int ph = pi & 3;
  const int panel = ph * 4 + (x8 >> 1);
  const int rb = (foreign ? 0 : 8) + 4 * (x8 & 1) + (pi >> 2);
  const int dst = dst_of(rank, x8);
  const bool xlink = ((dst ^ rank) & 4) != 0;
  const int src = wave;
  const __amdgpu_buffer_rsrc_t r_me = mk_rsrc((char*)me, WS_USED);
  const __amdgpu_buffer_rsrc_t r_out = mk_rsrc(p_out, OUT_BYTES);
  const __amdgpu_buffer_rsrc_t r_d = mk_rsrc(p_dst, WS_USED);
  const __amdgpu_buffer_rsrc_t r_x = mk_rsrc(p_x, IN_BYTES);
  const bool late_w = foreign && ph == 0 && !xlink;
  const bool late_h = foreign && ph == 1 && xlink;
  const bool late = foreign && ph < 2;
  const int lbl2 = 3 + (x8 & 3);
  const int ldst = late_h ? dst_of(rank, lbl2) : dst;
  auto mark_of = [&](int ln) {
    return (unsigned)(((64 * (dst & 3) + 16 * (pi & 3) + 2 * wave + ((ln >> 4) & 1)) * GN + (dst >> 2) * 1024 +
                       ((ln >> 5) * 4 + (pi >> 2)) * 64 + (ln & 15) * 4) * 4);
  };
  unsigned call, par, tag, cw;
  auto set_call = [&](unsigned call_v) {
    call = __builtin_amdgcn_readfirstlane(call_v) + 1u;
    par = call & 1u; tag = ((call + 1u) >> 1) & 1u;
    cw = call << 10;
  };
  auto stream_time = [&](unsigned d, bool& known) {
    const unsigned long long te = __hip_atomic_load((gu64*)(me + TE_OFF + d * 128u), __ATOMIC_RELAXED, __HIP_MEMORY_SCOPE_AGENT);
    known = __builtin_amdgcn_readfirstlane((unsigned)(te >> 32)) == call;
    return (unsigned)__builtin_amdgcn_readfirstlane((unsigned)te);
  };
  auto max_only = [&](int it, const float (&d)[8]) {
    float m = max3abs(0.f, d[0], d[1]);
    m = max3abs(m, d[2], d[3]); m = max3abs(m, d[4], d[5]); m = max3abs(m, d[6], d[7]);
    __hip_atomic_fetch_max(&sctl[16 * it + l15], __float_as_uint(m), __ATOMIC_RELAXED, __HIP_MEMORY_SCOPE_WORKGROUP);
  };

  if (__builtin_expect(!foreign, 1)) {
    if (wave == 0) { sctl[lane] = 0; sctl[64 + lane] = 0; WAIT_LGKM0(); }
    __builtin_amdgcn_s_barrier();
    float v[2][8];
    const __amdgpu_buffer_rsrc_t r_w = mk_rsrc(p_w, IN_BYTES);
    const unsigned wo = (unsigned)((32 * wave + 4 * kg) * GN + (dst >> 2) * 1024 + pi * 16 + l15) * 4u;
    auto phase_of = [&](int it) { return (unsigned)(xlink ? (it == 1 ? 0 : it == 2 ? 3 : it - 2) : it - 1); };
    auto load_w = [&](int it, float (&d)[8]) {
      const unsigned o = wo + phase_of(it) * 1024u;
#pragma unroll
      for (int j = 0; j < 8; ++j)
        d[j] = __builtin_bit_cast(float, __builtin_amdgcn_raw_buffer_load_b32(r_w, o, (16 * (j >> 2) + (j & 3)) * GN * 4, 0));
    };
    load_w(1, v[1]);
    unsigned grid;
    {
      const unsigned now = now256();
      grid = now - ((now & 0x3fffffffu) + 16u * SLOT256 - (unsigned)pi * SLOT256) % (16u * SLOT256);
      asm volatile("" : "+s"(grid));
    }
    __builtin_amdgcn_sched_barrier(0);
    {
      float m = max3abs(0.f, xlo.x, xlo.y);
      m = max3abs(m, xlo.z, xlo.w); m = max3abs(m, xhi.x, xhi.y); m = max3abs(m, xhi.z, xhi.w);
      __hip_atomic_fetch_max(&sctl[lane >> 2], __float_as_uint(m), __ATOMIC_RELAXED, __HIP_MEMORY_SCOPE_WORKGROUP);
      char* t = smem + LDS_PART + wave * 8192 + ((lane >> 2) + 17 * (lane & 3)) * 16;
      *(f4*)t = xlo; *(f4*)(t + 2048) = xhi;
    }
    WAIT_LGKM0();
    __builtin_amdgcn_s_barrier();
    COMPILER_FENCE();
    {
      const char* t = smem + LDS_PART + wave * 8192 + (l15 + 17 * kg) * 16;
      const f4 lo = *(const f4*)t, hi = *(const f4*)(t + 2048);
      v[0][0] = lo.x; v[0][1] = lo.y; v[0][2] = lo.z; v[0][3] = lo.w; v[0][4] = hi.x; v[0][5] = hi.y; v[0][6] = hi.z; v[0][7] = hi.w;
    }
    set_call(call_v);
    auto max_item = [&](int it, const float (&d)[8]) { max_only(it, d); lds_arrive(&sctl[C_ITEM + it], lane); };
    const unsigned xa_d = XA_OFF + par * XA_PAR + (unsigned)rank * XA_SRC + (unsigned)(wave * 32 + pi) * 1024u + l16;
    const unsigned wb_d = WB_OFF + par * WB_PAR + (unsigned)rank * WB_SRC + (unsigned)(wave * 64 + pi) * 1024u + l16;
    const unsigned ex_d = EX_OFF + (par * 8 + rank) * 2048u + pi * 64u, ew_d = EW_OFF + (par * 8 + rank) * 4096u + pi * 64u;
    const int ri = pi - (16 - 4 * RQ);
    const bool send3 = !xlink || ri < 0 || ri / RQ == (dst & 3);
    const int idx3 = ri < 0 ? pi : 16 - 4 * RQ + ri % RQ;
#pragma unroll
    for (int it = 0; it < 5; ++it) {
      if (it == 4 && !xlink) break;
      if (it) lds_wait(&sctl[C_ITEM + it], NW);
      const int e = exp_of(__hip_atomic_load(&sctl[16 * it + l15], __ATOMIC_RELAXED, __HIP_MEMORY_SCOPE_WORKGROUP));
      u4 f = pack8(v[it & 1], -e, tag);
      asm volatile("" : "+v"(f));
      if (it == 0) {
        if (wave == 0) {
          unsigned long long t;
          asm volatile("s_memrealtime %0\n\ts_waitcnt lgkmcnt(0)" : "=s"(t) : "v"(f));
          const unsigned now = (unsigned)t << 8;
          while ((int)(now - grid) > 0) grid += 16u * SLOT256;
          grid |= 1u;
          if (lane == 0) {
            __hip_atomic_store(&sctl[C_GRID], grid, __ATOMIC_RELAXED, __HIP_MEMORY_SCOPE_WORKGROUP);
            if (pi == 0)
              __hip_atomic_store((gu64*)(me + TE_OFF + dst * 128u), ((unsigned long long)call << 32) | (now + BASE_AHEAD * SLOT256),
                                 __ATOMIC_RELAXED, __HIP_MEMORY_SCOPE_AGENT);
          }
        } else {
          lds_wait(&sctl[C_GRID], 1u);
          grid = __builtin_amdgcn_readfirstlane(__hip_atomic_load(&sctl[C_GRID], __ATOMIC_RELAXED, __HIP_MEMORY_SCOPE_WORKGROUP));
        }
        if (wave >= 4) max_item(1, v[1]);
      }
      int slot = it * 128 + wave * 16;
      if (xlink) { if (it == 2) slot = 256 + wave * (int)SHORT_N + idx3 - pi; else if (it > 2) slot += (int)SHORT_LEN - 128; }
      wait_until(grid + (unsigned)slot * SLOT256);
      if (it != 2 || send3) { st128(r_d, it == 0 ? xa_d : wb_d + phase_of(it) * 16384u, f); }
      if (wave == 1 && lane < 16) st32(r_d, lane * 4, it == 0 ? ex_d : ew_d + phase_of(it) * 1024u, cw | (unsigned)(e + 512));
      if (it == 0) {
        if (xlink) {
          const unsigned mark = mark_of(lane);
          st128(r_out, mark, u4{EMPTY, EMPTY, EMPTY, EMPTY});
          st128(r_out, mark + 2048u, u4{EMPTY, EMPTY, EMPTY, EMPTY});
        }
      }
      if (it == 1 && wave == 0 && lane == 0) __hip_atomic_store(ctr, call, __ATOMIC_RELAXED, __HIP_MEMORY_SCOPE_AGENT);
      if (it + 2 < 4 || (it == 2 && xlink)) {
        load_w(it + 2, v[it & 1]);
        if (it == 0) { if (wave < 4) max_item(1, v[1]); WAIT_VM0(); }
        max_item(it + 2, v[it & 1]);
        if (it == 0) {
          bool known;
          const unsigned g0 = stream_time(dst, known);
          if (known) grid = g0 + (unsigned)pi * SLOT256;
        }
      }
    }
    wait_until(grid + ((xlink ? X_XO : I_RELAY) - 16u) * SLOT256 + 2048u);
  } else {
    if (wave == 0) { sctl[lane] = 0; sctl[64 + lane] = 0; WAIT_LGKM0(); }
    set_call(call_v);
    WAIT_VM0();
    __builtin_amdgcn_s_barrier();
    if (wave == 0 && lane == 0) __hip_atomic_store(ctr, call, __ATOMIC_RELAXED, __HIP_MEMORY_SCOPE_AGENT);
    if (late) {
      __builtin_amdgcn_s_sleep(32);
      const unsigned xo = (unsigned)(l15 * KS + 32 * wave + 4 * kg) * 4u;
      float v[4][8];
      if (late_w) {
        const __amdgpu_buffer_rsrc_t r_w = mk_rsrc(p_w, IN_BYTES);
        const unsigned wo = (unsigned)((32 * wave + 4 * kg) * GN + (dst >> 2) * 1024 + (48 + (pi >> 2)) * 16 + l15) * 4u;
#pragma unroll
        for (int u = 0; u < 4; ++u)
#pragma unroll
          for (int j = 0; j < 8; ++j)
            v[u][j] = __builtin_bit_cast(float, __builtin_amdgcn_raw_buffer_load_b32(r_w, wo + u * 256, (16 * (j >> 2) + (j & 3)) * GN * 4, 0));
      } else {
#pragma unroll
        for (int u = 0; u < 4; ++u) {
          const int row = 256 * ldst + 16 * ((pi >> 2) + 4 * u);
          const f4 lo = __builtin_bit_cast(f4, __builtin_amdgcn_raw_buffer_load_b128(r_x, xo, row * KS * 4, 0));
          const f4 hi = __builtin_bit_cast(f4, __builtin_amdgcn_raw_buffer_load_b128(r_x, xo + 64, row * KS * 4, 0));
          v[u][0] = lo.x; v[u][1] = lo.y; v[u][2] = lo.z; v[u][3] = lo.w; v[u][4] = hi.x; v[u][5] = hi.y; v[u][6] = hi.z; v[u][7] = hi.w;
        }
      }
#pragma unroll
      for (int u = 0; u < 4; ++u) max_only(u, v[u]);
      WAIT_LGKM0();
      __builtin_amdgcn_s_barrier();
      COMPILER_FENCE();
#pragma unroll
      for (int u = 0; u < 4; ++u) {
        const int e = exp_of(__hip_atomic_load(&sctl[16 * u + l15], __ATOMIC_RELAXED, __HIP_MEMORY_SCOPE_WORKGROUP));
        *(u4*)(smem + LDS_XO + (wave * 4 + u) * 1024 + l16) = pack8(v[u], -e, tag);
      }
    }
  }

  Waiter wt; wt.tmo = (gu32*)(me + TMO_OFF); wt.call = call; wt.start();
  const unsigned wb_base = WB_OFF + par * WB_PAR + (unsigned)src * WB_SRC + (unsigned)(panel * 4) * 1024u;
  const unsigned xa_base = XA_OFF + par * XA_PAR + (unsigned)src * XA_SRC + (unsigned)(rb * 2) * 1024u;
  const unsigned tagm = tag ? 0u : ~0u;
  int* sc = (int*)(smem + LDS_SC + wave * 512);
  const unsigned hoff = (unsigned)((src + (lane & 3) * 2) & 7) * 128u + 112u;

  u4 A[8][2], B[8][4];
  f4 acc[2][4];
#pragma unroll
  for (int mi = 0; mi < 2; ++mi)
#pragma unroll
    for (int ni = 0; ni < 4; ++ni) acc[mi][ni] = f4{0.f, 0.f, 0.f, 0.f};
#define PROBE_A(kb) do { for (;;) { const unsigned pv = ld32(r_me, (lane & 1) * 1024u + hoff, xa_base + (kb) * 32768u); \
      if (__all((pv ^ tagm) & 1u) || wt.expired()) break; } COMPILER_FENCE(); } while (0)
#define PROBE_B(kb) do { for (;;) { const unsigned pv = ld32(r_me, (lane & 3) * 1024u + hoff, wb_base + (kb) * 65536u); \
      if (__all((pv ^ tagm) & 1u) || wt.expired()) break; } COMPILER_FENCE(); } while (0)
#define ISSUE_A(k0, k1) _Pragma("unroll") for (int kb = (k0); kb < (k1); ++kb) _Pragma("unroll") for (int mi = 0; mi < 2; ++mi) \
      A[kb][mi] = ld128(r_me, l16 + mi * 1024u, xa_base + kb * 32768u)
#define ISSUE_B(k0, k1) _Pragma("unroll") for (int kb = (k0); kb < (k1); ++kb) _Pragma("unroll") for (int ni = 0; ni < 4; ++ni) \
      B[kb][ni] = ld128(r_me, l16 + ni * 1024u, wb_base + kb * 65536u)
#define CHECK_A(k0, k1) _Pragma("unroll") for (int kb = (k0); kb < (k1); ++kb) _Pragma("unroll") for (int mi = 0; mi < 2; ++mi) ok &= A[kb][mi].x ^ tagm
#define CHECK_B(k0, k1) _Pragma("unroll") for (int kb = (k0); kb < (k1); ++kb) _Pragma("unroll") for (int ni = 0; ni < 4; ++ni) ok &= B[kb][ni].x ^ tagm
#define MMA(k0, k1) _Pragma("unroll") for (int kb = (k0); kb < (k1); ++kb) _Pragma("unroll") for (int mi = 0; mi < 2; ++mi) \
      _Pragma("unroll") for (int ni = 0; ni < 4; ++ni) \
        acc[mi][ni] = __builtin_amdgcn_mfma_f32_16x16x32_f16(__builtin_bit_cast(h8, B[kb][ni]), __builtin_bit_cast(h8, A[kb][mi]), acc[mi][ni], 0, 0, 0)
  auto ld_ex = [&]() { return ld32(r_me, (lane & 31) * 4, EX_OFF + (par * 8 + src) * 2048u + rb * 128u); };
  auto ld_ew = [&]() { return ld32(r_me, lane * 4, EW_OFF + (par * 8 + src) * 4096u + panel * 256u); };

  if (foreign) {
    PROBE_A(7);
    for (;;) {
      unsigned ok = 1u;
      ISSUE_A(0, 8);
      const unsigned exv = ld_ex();
      CHECK_A(0, 8);
      sc[64 + lane] = (int)(exv & 1023u) - 1024;
      if (__all((ok & 1u) && (exv ^ cw) < 1024u) || wt.expired()) break;
    }
    if (ph == 3 && !xlink && dst != rank) {
      const unsigned fo = WB_OFF + par * WB_PAR + (unsigned)((((rank >> 2) ^ 1) << 2) | (pi >> 2)) * WB_SRC +
                          (unsigned)(wave * 64 + 64 - 4 * RQ + (rank & 3) * RQ) * 1024u + l16;
      bool known;
      const unsigned te = stream_time(dst, known);
      if (known) wait_until(te + (256u + SHORT_LEN) * SLOT256);
      u4 f[RQ];
      for (;;) {
        unsigned ok = 1u;
#pragma unroll
        for (int u = 0; u < RQ; ++u) { f[u] = ld128(r_me, fo + u * 1024u, 0); }
#pragma unroll
        for (int u = 0; u < RQ; ++u) ok &= f[u].x ^ tagm;
        if (__all(ok & 1u) || wt.expired()) break;
      }
#pragma unroll
      for (int u = 0; u < RQ; ++u) {
        if (known) wait_until(te + (I_RELAY + (unsigned)(wave * 4 * RQ + u * 4 + (pi >> 2))) * SLOT256);
        st128(r_d, fo + u * 1024u, f[u]);
      }
    }
    PROBE_B(SPLITK - 1);
    for (;;) {
      unsigned ok = 1u;
      ISSUE_B(0, SPLITK);
      CHECK_B(0, SPLITK);
      if (__all(ok & 1u) || wt.expired()) break;
    }
    MMA(0, SPLITK);
    PROBE_B(7);
    for (;;) {
      unsigned ok = 1u;
      ISSUE_B(SPLITK, 8);
      const unsigned ewv = ld_ew();
      CHECK_B(SPLITK, 8);
      sc[lane] = (int)(ewv & 1023u);
      if (__all((ok & 1u) && (ewv ^ cw) < 1024u) || wt.expired()) break;
    }
    MMA(SPLITK, 8);
  } else {
    PROBE_B(7);
    bool a1, a2;
    {
      unsigned ok = 1u;
      ISSUE_B(0, 8);
      const unsigned ewv = ld_ew();
      ISSUE_A(0, 8);
      const unsigned exv = ld_ex();
      CHECK_B(0, 8);
      sc[lane] = (int)(ewv & 1023u);
      bool bok = __all((ok & 1u) && (ewv ^ cw) < 1024u);
      ok = 1u;
      CHECK_A(0, SPLITA);
      a1 = __all(ok & 1u);
      ok = 1u;
      CHECK_A(SPLITA, 8);
      sc[64 + lane] = (int)(exv & 1023u) - 1024;
      a2 = __all((ok & 1u) && (exv ^ cw) < 1024u);
      while (!bok && !wt.expired()) {
        ok = 1u;
        ISSUE_B(0, 8);
        const unsigned ew2 = ld_ew();
        CHECK_B(0, 8);
        sc[lane] = (int)(ew2 & 1023u);
        bok = __all((ok & 1u) && (ew2 ^ cw) < 1024u);
      }
    }
    if (!a1) {
      PROBE_A(SPLITA - 1);
      for (;;) {
        unsigned ok = 1u;
        ISSUE_A(0, SPLITA);
        CHECK_A(0, SPLITA);
        if (__all(ok & 1u) || wt.expired()) break;
      }
    }
    MMA(0, SPLITA);
    if (!a2) {
      for (;;) {
        unsigned ok = 1u;
        ISSUE_A(SPLITA, 8);
        const unsigned exv = ld_ex();
        CHECK_A(SPLITA, 8);
        sc[64 + lane] = (int)(exv & 1023u) - 1024;
        if (__all((ok & 1u) && (exv ^ cw) < 1024u) || wt.expired_fast()) break;
      }
    }
    MMA(SPLITA, 8);
  }

  int ln = lane;
  asm volatile("" : "+v"(ln));
  const int e15 = ln & 15, ekg = ln >> 4;
  {
    WAIT_LGKM0();
    int exv[2];
    i4 ewv[4];
#pragma unroll
    for (int mi = 0; mi < 2; ++mi) exv[mi] = sc[64 + mi * 16 + e15];
#pragma unroll
    for (int ni = 0; ni < 4; ++ni) ewv[ni] = *(const i4*)(sc + ni * 16 + ekg * 4);
#pragma unroll
    for (int mi = 0; mi < 2; ++mi)
#pragma unroll
      for (int ni = 0; ni < 4; ++ni) {
        f4 o;
        o.x = __builtin_amdgcn_ldexpf(acc[mi][ni].x, exv[mi] + ewv[ni].x);
        o.y = __builtin_amdgcn_ldexpf(acc[mi][ni].y, exv[mi] + ewv[ni].y);
        o.z = __builtin_amdgcn_ldexpf(acc[mi][ni].z, exv[mi] + ewv[ni].z);
        o.w = __builtin_amdgcn_ldexpf(acc[mi][ni].w, exv[mi] + ewv[ni].w);
        const int q = ni * 4 + ekg;
        *(f4*)(smem + LDS_PART + wave * 8192 + ((mi * 4 + ni) * 64 + (ln & 48) + (e15 ^ q)) * 16) = o;
      }
  }
  if (wt.gave_up && lane == 0) __hip_atomic_store(&sctl[C_DEAD], 1u, __ATOMIC_RELAXED, __HIP_MEMORY_SCOPE_WORKGROUP);
  WAIT_LGKM0();
  __builtin_amdgcn_s_barrier();
  COMPILER_FENCE();
  const bool dead = __hip_atomic_load(&sctl[C_DEAD], __ATOMIC_RELAXED, __HIP_MEMORY_SCOPE_WORKGROUP) != 0;
  const int m = wave * 4 + ekg, q = e15;
  const int idx = ((m >> 4) * 4 + (q >> 2)) * 64 + (q & 3) * 16 + ((m & 15) ^ q);
  f4 s = *(const f4*)(smem + LDS_PART + idx * 16);
#pragma unroll
  for (int w = 1; w < NW; ++w) s += *(const f4*)(smem + LDS_PART + w * 8192 + idx * 16);
  u4 o;
#pragma unroll
  for (int e = 0; e < 4; ++e) {
    const float y = s[e];
    const float c = __builtin_fminf(__builtin_fmaxf(y, -60.f), 60.f);
    const unsigned r = __float_as_uint(y * __builtin_amdgcn_rcpf(1.f + __expf(-c)));
    o[e] = (dead || r == EMPTY) ? QNAN : r;
  }
  if (foreign) {
    const unsigned owner = (unsigned)((((rank >> 2) ^ 1) << 2) | (rb >> 1));
    const unsigned row = (unsigned)(64 * (rank & 3) + 32 * (rb & 1) + m);
    const unsigned col = (unsigned)((rank >> 2) * 1024 + panel * 64 + q * 4);
    const __amdgpu_buffer_rsrc_t r_ow = mk_rsrc(a.out[owner & 7], OUT_BYTES);
    asm volatile("" : "+v"(o));
    bool known_o;
    const int oc = xlink ? 0 : 1, oi = x8 >= (xlink ? 2 : 5) ? 1 : 0;
    const int ok_ = (ph < 3 ? oc * 3 + ph : 6 + oc) * 4 + oi * 2 + (rb & 1);
    const unsigned t_o = stream_time(owner, known_o) + (X_OUT + (unsigned)(ok_ * 8 + wave)) * SLOT256;
    bool o_sent = false;
    if (late && !(ph == 1 && (wave >= XSW) != late_h)) {
      const __amdgpu_buffer_rsrc_t r_l = mk_rsrc(a.ws_dst[late_h ? lbl2 : x8], WS_USED);
      bool known_d;
      const unsigned t_x = stream_time(ldst, known_d) + (((xlink && !late_h) ? X_XO : late_w ? I_XO - 128u : I_XO) + (unsigned)(wave * 16 + (pi >> 2))) * SLOT256;
      const unsigned xa_d = ln * 16u + (late_w ? WB_OFF + par * WB_PAR + (unsigned)rank * WB_SRC + (unsigned)(wave * 64 + 48 + (pi >> 2)) * 1024u
                                               : XA_OFF + par * XA_PAR + (unsigned)rank * XA_SRC + (unsigned)(wave * 32 + 16 + (pi >> 2)) * 1024u);
      const unsigned sc_d = (pi >> 2) * 64u + (late_w ? EW_OFF + (par * 8 + rank) * 4096u + 3072u : EX_OFF + (par * 8 + rank) * 2048u + 1024u);
      u4 f[4];
#pragma unroll
      for (int u = 0; u < 4; ++u) f[u] = *(const u4*)(smem + LDS_XO + (wave * 4 + u) * 1024 + ln * 16);
      const unsigned ew = cw | (unsigned)(exp_of(sctl[ln]) + 512);
      if (known_o && known_d && (int)(t_o - t_x) < 0) {
        wait_until(t_o);
        st128(r_ow, (row * GN + col) * 4u, o);
        o_sent = true;
      }
#pragma unroll
      for (int u = 0; u < 4; ++u) {
        if (known_d) wait_until(t_x + (unsigned)(4 * u) * SLOT256);
        st128(r_l, xa_d + u * 4096u, f[u]);
      }
      if (wave == 1) st32(r_l, e15 * 4 + ekg * 256, sc_d, ew);
    }
    if (!o_sent) {
      if (known_o) wait_until(t_o);
      st128(r_ow, (row * GN + col) * 4u, o);
    }
  } else {
    const unsigned oo = (unsigned)(((32 * (rb - 8) + m) * GN + (rank >> 2) * 1024 + panel * 64 + q * 4) * 4);
    st128(r_out, oo, o);
    if (xlink) {
      const unsigned mark = mark_of(ln);
      for (;;) {
        const u4 t = ld128(r_out, mark, 0), u = ld128(r_out, mark + 2048u, 0);
        if (__all(t.x != EMPTY && t.y != EMPTY && t.z != EMPTY && t.w != EMPTY && u.x != EMPTY && u.y != EMPTY && u.z != EMPTY && u.w != EMPTY)) break;
        if (wt.expired_fast()) {
          WAIT_VM0();
          st32(r_out, oo, 0, QNAN);
          break;
        }
      }
    }
  }
}

extern "C" void kernel_launch(int rank, int nranks, ncclComm_t comm, hipStream_t stream, void* const* d_in,
                              const int* in_sizes, int n_in, void* d_out, int out_size, const int* in_dtypes,
                              int out_dtype, void* const* d_out_all, void* const* d_ws_all, size_t ws_bytes) {
  if (nranks != NR || (unsigned)rank >= (unsigned)NR || ws_bytes < WS_USED || n_in < 2 || in_sizes[0] != 2048 * KS ||
      in_sizes[1] != KS * GN || out_size != 256 * GN) {
    fprintf(stderr, "unexpected problem shape\n");
    return;
  }
  Args a;
  a.x = (const float*)d_in[0];
  a.w = (const float*)d_in[1];
  for (int p = 0; p < NR; ++p) {
    const int d = dst_of(rank, p) & 7;
    a.xf[p] = a.x + (size_t)(1024 * ((d >> 2) ^ 1) + 64 * (d & 3)) * KS;
  }
  for (int p = 0; p < NR; ++p) { a.ws_dst[p] = (char*)d_ws_all[dst_of(rank, p) & 7]; a.out[p] = (float*)d_out_all[p]; }
  a.out[rank] = (float*)d_out;
  a.me = (char*)d_ws_all[rank];
  a.out_me = (float*)d_out;
  a.rank = rank;
  fused_gemm_rs_silu<<<NWG, NTHREADS, 0, stream>>>(a);
}
```
